```python
import numpy as np

kernel_name = "gpu_ag_gemm_m4096_k4096_n4096_e4m3e4m3_dqrelu"


ABSMAX_RTOL = 0.03

COMPUTE_DTYPE = "float32"
EPILOGUE = "dequant_relu"
IN_QUANT = None
OUT_QUANT = None
IN_FP8 = "e4m3e4m3"
IN_DTYPES = {"x": "e4m3", "w_mat": "e4m3",
             "scale_x": "float32", "scale_w": "float32"}
WORLD_SIZE = 8

def _snap_fp8(x, mant_bits, max_abs, min_normal):
    a = np.abs(x.astype(np.float64))
    _m, _e = np.frexp(a)
    q = np.where(a >= min_normal, np.ldexp(1.0, _e - 1 - mant_bits),
                 min_normal * 2.0 ** -mant_bits)
    snapped = np.minimum(np.round(a / q) * q, max_abs)
    return (np.sign(x) * snapped).astype(np.float32)

def setup_inputs():

    x = _snap_fp8(np.random.randn(4096, 4096).astype(np.float32), 3, 448.0, 2.0 ** -6)
    w_mat = _snap_fp8(np.random.randn(4096, 4096).astype(np.float32), 3, 448.0, 2.0 ** -6)
    scale_x = (0.02 * (1.0 + np.random.rand(1))).astype(np.float32)
    scale_w = (0.02 * (1.0 + np.random.rand(1))).astype(np.float32)
    return {"x": x, "w_mat": w_mat, "scale_x": scale_x, "scale_w": scale_w}

def reference(x, w_mat, scale_x, scale_w):
    acc = (x @ w_mat).astype(np.float32)
    y = (acc * (scale_x * scale_w)).astype(np.float32)
    return np.maximum(y, 0.0).astype(np.float32)
```

```cpp
#include <hip/hip_runtime.h>
#include <rccl/rccl.h>
#include <cstdio>
#include <cstdint>

typedef unsigned u32x4 __attribute__((ext_vector_type(4)));
typedef int i32x8 __attribute__((ext_vector_type(8)));
typedef float f32x4 __attribute__((ext_vector_type(4)));
typedef float f32x2 __attribute__((ext_vector_type(2)));

namespace {

constexpr int NR = 8;
constexpr int KD = 4096;
constexpr int MS = 512;
constexpr int NS = 512;
constexpr int NBLK = 32;
constexpr int BLK_BYTES = 16 * KD;
constexpr int SHARD_BYTES = MS * KD;
#define NCLASS 4
constexpr int NSTEP = NBLK / NCLASS;
constexpr int NPUSH = 7 * NCLASS;
constexpr int NPUSH_WG = 32;
constexpr int NGRID = 256;
constexpr int NTHREADS = 512;

constexpr size_t CNT_OFF = 0;
constexpr size_t TMO_OFF = 32768;
constexpr size_t FLAG_OFF = 65536;
constexpr size_t DBG_OFF = 131072;
constexpr size_t STAGE_OFF = 1u << 20;
constexpr size_t SLOT_BYTES = (size_t)NR * SHARD_BYTES;
constexpr size_t WS_NEED = STAGE_OFF + 2 * SLOT_BYTES;

constexpr unsigned long long DEADLINE_TICKS = 40000000ull;

struct Params {
    const uint8_t* x;
    const uint8_t* w;
    const float* sx;
    const float* sw;
    float* out;
    uint8_t* ws[NR];
    int rank;
    int pad;
};

__device__ __forceinline__ unsigned ld_sys(const unsigned* p) {
    return __hip_atomic_load(p, __ATOMIC_RELAXED, __HIP_MEMORY_SCOPE_SYSTEM);
}
__device__ __forceinline__ unsigned ld_agent(const unsigned* p) {
    return __hip_atomic_load(p, __ATOMIC_RELAXED, __HIP_MEMORY_SCOPE_AGENT);
}

#define STAMP(idx) do {} while (0)

#define LOAD8(R, ptr) asm volatile( \
    "global_load_dwordx4 %0, %8, off\n\t" \
    "global_load_dwordx4 %1, %8, off offset:64\n\t" \
    "global_load_dwordx4 %2, %8, off offset:128\n\t" \
    "global_load_dwordx4 %3, %8, off offset:192\n\t" \
    "global_load_dwordx4 %4, %8, off offset:256\n\t" \
    "global_load_dwordx4 %5, %8, off offset:320\n\t" \
    "global_load_dwordx4 %6, %8, off offset:384\n\t" \
    "global_load_dwordx4 %7, %8, off offset:448" \
    : "=&v"(R[0]), "=&v"(R[1]), "=&v"(R[2]), "=&v"(R[3]), "=&v"(R[4]), "=&v"(R[5]), "=&v"(R[6]), "=&v"(R[7]) \
    : "v"(ptr) : "memory")

#define STBITS "sc0 sc1"
#define STORE8(R, ptr) asm volatile( \
    "global_store_dwordx4 %8, %0, off offset:-4096 " STBITS "\n\t" \
    "global_store_dwordx4 %8, %1, off offset:-3072 " STBITS "\n\t" \
    "global_store_dwordx4 %8, %2, off offset:-2048 " STBITS "\n\t" \
    "global_store_dwordx4 %8, %3, off offset:-1024 " STBITS "\n\t" \
    "global_store_dwordx4 %8, %4, off " STBITS "\n\t" \
    "global_store_dwordx4 %8, %5, off offset:1024 " STBITS "\n\t" \
    "global_store_dwordx4 %8, %6, off offset:2048 " STBITS "\n\t" \
    "global_store_dwordx4 %8, %7, off offset:3072 " STBITS "\n\t" \
    "s_nop 1" \
    :: "v"(R[0]), "v"(R[1]), "v"(R[2]), "v"(R[3]), "v"(R[4]), "v"(R[5]), "v"(R[6]), "v"(R[7]), "v"(ptr) : "memory")

#define WAITV(n) asm volatile("s_waitcnt vmcnt(" #n ")" ::: "memory")
#define BAR() __builtin_amdgcn_s_barrier()

__device__ __forceinline__ unsigned draw_call(const Params& p) {
    unsigned* cnt = (unsigned*)(p.ws[p.rank] + CNT_OFF + (size_t)blockIdx.x * 128) + (threadIdx.x >> 6);
    unsigned c = 0;
    if ((threadIdx.x & 63) == 0) c = __hip_atomic_fetch_add(cnt, 1u, __ATOMIC_RELAXED, __HIP_MEMORY_SCOPE_AGENT) + 1u;
    return (unsigned)__builtin_amdgcn_readfirstlane((int)c);
}

__device__ __forceinline__ void pusher(const Params& p) {
    const int id = blockIdx.x;
    if (id >= NPUSH) return;
    const int tid = threadIdx.x, w = tid >> 6, l = tid & 63, r = l & 15, q = l >> 4;
    const int dd = id % 7, cls = id / 7;
    const unsigned peer = (unsigned)(p.rank + 1 + dd) & 7u;
    STAMP(0);
    const uint8_t* src = p.x + (size_t)(16 * cls + r) * KD + 512 * w + 16 * q;
    constexpr size_t SRC_STEP = (size_t)16 * NCLASS * KD;
    u32x4 A[8], B[8];
    LOAD8(A, src);
    if (NSTEP > 1) LOAD8(B, src + SRC_STEP);
    const unsigned call = draw_call(p);
    uint8_t* pw = p.ws[peer];
    uint8_t* dst = pw + STAGE_OFF + (size_t)(call & 1u) * SLOT_BYTES + (size_t)p.rank * SHARD_BYTES +
                   (size_t)cls * BLK_BYTES + w * 8192 + l * 16 + 4096;
    constexpr size_t DST_STEP = (size_t)NCLASS * BLK_BYTES;
    unsigned* flag = (unsigned*)(pw + FLAG_OFF + (size_t)(p.rank * 16 + cls) * 128);
    const unsigned base = (call - 1u) * NSTEP;
    WAITV(0);
    STAMP(1);
#define FLAG(t) do { BAR(); if (tid == 0) __hip_atomic_store(flag, base + (t) + 1u, __ATOMIC_RELAXED, __HIP_MEMORY_SCOPE_SYSTEM); STAMP(2 + (t)); } while (0)
#pragma unroll
    for (int t = 0; t < NSTEP; ++t) {
        if (t >= 2) {
            if (t + 1 < NSTEP) WAITV(16); else WAITV(8);
            FLAG(t - 2);
        }
        uint8_t* d = dst + (size_t)t * DST_STEP;
        if (t & 1) STORE8(B, d); else STORE8(A, d);
        if (t + 2 < NSTEP) {
            const uint8_t* s = src + (size_t)(t + 2) * SRC_STEP;
            if (t & 1) LOAD8(B, s); else LOAD8(A, s);
        }
    }
    if (NSTEP >= 2) { WAITV(8); FLAG(NSTEP - 2); }
    WAITV(0);
    FLAG(NSTEP - 1);
#undef FLAG
}

__device__ __forceinline__ void transpose4(unsigned a, unsigned b, unsigned c, unsigned d,
                                           unsigned& o0, unsigned& o1, unsigned& o2, unsigned& o3) {
    const unsigned t0 = __builtin_amdgcn_perm(b, a, 0x05010400u);
    const unsigned t1 = __builtin_amdgcn_perm(b, a, 0x07030602u);
    const unsigned t2 = __builtin_amdgcn_perm(d, c, 0x05010400u);
    const unsigned t3 = __builtin_amdgcn_perm(d, c, 0x07030602u);
    o0 = __builtin_amdgcn_perm(t2, t0, 0x05040100u);
    o1 = __builtin_amdgcn_perm(t2, t0, 0x07060302u);
    o2 = __builtin_amdgcn_perm(t3, t1, 0x05040100u);
    o3 = __builtin_amdgcn_perm(t3, t1, 0x07060302u);
}

__device__ __forceinline__ i32x8 join8(u32x4 lo, u32x4 hi) {
    i32x8 v;
    v[0] = lo[0]; v[1] = lo[1]; v[2] = lo[2]; v[3] = lo[3];
    v[4] = hi[0]; v[5] = hi[1]; v[6] = hi[2]; v[7] = hi[3];
    return v;
}

struct Smem {
    float red[8][16][64];
    unsigned word[4];
};

__device__ __forceinline__ void gemm_block(const Params& p, Smem& sm, const u32x4 (&Af)[8], const i32x8 (&Bf)[4][4],
                                           int row0, int col0, float scale, bool bad) {
    const int tid = threadIdx.x, w = tid >> 6, l = tid & 63, c = l & 15, q = l >> 4;
    f32x4 acc[4];
#pragma unroll
    for (int i = 0; i < 4; ++i) acc[i] = f32x4{0.f, 0.f, 0.f, 0.f};
#pragma unroll
    for (int s = 0; s < 4; ++s) {
        const i32x8 a = join8(Af[2 * s], Af[2 * s + 1]);
#pragma unroll
        for (int i = 0; i < 4; ++i)
            acc[i] = __builtin_amdgcn_mfma_scale_f32_16x16x128_f8f6f4(a, Bf[s][i], acc[i], 0, 0, 0, 0, 0, 0);
    }
#pragma unroll
    for (int t = 0; t < 4; ++t)
        *(f32x4*)&sm.red[w][4 * q + t][4 * c] = f32x4{acc[0][t], acc[1][t], acc[2][t], acc[3][t]};
    __syncthreads();
    const int row = tid >> 5, c2 = (tid & 31) * 2;
    f32x2 v = *(const f32x2*)&sm.red[0][row][c2];
#pragma unroll
    for (int k = 1; k < 8; ++k) v += *(const f32x2*)&sm.red[k][row][c2];
    v *= scale;
    v[0] = fmaxf(v[0], 0.f);
    v[1] = fmaxf(v[1], 0.f);
    if (bad) v = f32x2{__builtin_nanf(""), __builtin_nanf("")};
    *(f32x2*)(p.out + (size_t)(row0 + row) * NS + col0 + c2) = v;
}

__device__ __forceinline__ void gemm_wg(const Params& p, Smem& sm) {
    const int tid = threadIdx.x, w = tid >> 6, l = tid & 63, r = l & 15, q = l >> 4;
    const int gid = blockIdx.x - NPUSH_WG, xcd = gid & 7, tt = gid >> 3;
    int g, j;
    if (tt < 24) { g = 3 * xcd + (tt >> 3); j = tt & 7; }
    else { g = 24 + (xcd >> 1); j = (xcd & 1) * 4 + (tt - 24); }
    STAMP(0);
    const unsigned call = draw_call(p);
    i32x8 Bf[4][4];
    {
        const uint8_t* wp = p.w + (size_t)(512 * w + 16 * q) * NS + 64 * j + 4 * r;
#pragma unroll
        for (int s = 0; s < 4; ++s) {
            unsigned t[32];
#pragma unroll
            for (int h = 0; h < 2; ++h)
#pragma unroll
                for (int k = 0; k < 16; ++k)
                    t[h * 16 + k] = *(const unsigned*)(wp + (size_t)(128 * s + 64 * h + k) * NS);
#pragma unroll
            for (int jj = 0; jj < 8; ++jj) {
                unsigned o0, o1, o2, o3;
                transpose4(t[4 * jj], t[4 * jj + 1], t[4 * jj + 2], t[4 * jj + 3], o0, o1, o2, o3);
                Bf[s][0][jj] = o0; Bf[s][1][jj] = o1; Bf[s][2][jj] = o2; Bf[s][3][jj] = o3;
            }
        }
    }
    const float scale = p.sx[0] * p.sw[0];
    const int col0 = 64 * j;
    STAMP(1);
    for (int lb = g; lb < NBLK; lb += 28) {
        const uint8_t* src = p.x + (size_t)(16 * lb + r) * KD + 512 * w + 16 * q;
        u32x4 Af[8];
#pragma unroll
        for (int k = 0; k < 8; ++k) Af[k] = *(const u32x4*)(src + 64 * k);
        gemm_block(p, sm, Af, Bf, p.rank * MS + 16 * lb, col0, scale, false);
        __syncthreads();
    }
    STAMP(2);
    const int cls = g & 3;
    const unsigned srcr = (unsigned)(p.rank + 1 + (g >> 2)) & 7u;
    uint8_t* ws = p.ws[p.rank];
    unsigned* tmo = (unsigned*)(ws + TMO_OFF);
    constexpr int PER = NCLASS / 4 > 0 ? NCLASS / 4 : 1;
    const uint8_t* stage = ws + STAGE_OFF + (size_t)(call & 1u) * SLOT_BYTES + (size_t)srcr * SHARD_BYTES +
                           w * 8192 + l * 16 + 4096;
    for (int t = 0; t < 8; ++t) {
        const int b = 4 * t + cls;
        const int pc = b % NCLASS, ps = b / NCLASS;
        if (tid == 0) {
            const unsigned* flag = (const unsigned*)(ws + FLAG_OFF + (size_t)(srcr * 16 + pc) * 128);
            const unsigned seq = (call - 1u) * NSTEP + ps + 1u;
            const unsigned long long t0 = __builtin_amdgcn_s_memrealtime();
            unsigned it = 0;
            while (ld_sys(flag) < seq) {
                if ((++it & 15u) == 0u &&
                    (ld_agent(tmo) == call || __builtin_amdgcn_s_memrealtime() - t0 > DEADLINE_TICKS)) {
                    __hip_atomic_store(tmo, call, __ATOMIC_RELAXED, __HIP_MEMORY_SCOPE_AGENT);
                    break;
                }
                __builtin_amdgcn_s_sleep(2);
            }
            sm.word[0] = (ld_agent(tmo) == call) ? 1u : 0u;
        }
        __syncthreads();
        STAMP(3 + 2 * t);
        const bool bad = sm.word[0] != 0u;
        u32x4 Af[8];
        const uint8_t* a = stage + (size_t)b * BLK_BYTES;
        asm volatile(
            "global_load_dwordx4 %0, %8, off offset:-4096 sc0 sc1\n\t"
            "global_load_dwordx4 %1, %8, off offset:-3072 sc0 sc1\n\t"
            "global_load_dwordx4 %2, %8, off offset:-2048 sc0 sc1\n\t"
            "global_load_dwordx4 %3, %8, off offset:-1024 sc0 sc1\n\t"
            "global_load_dwordx4 %4, %8, off sc0 sc1\n\t"
            "global_load_dwordx4 %5, %8, off offset:1024 sc0 sc1\n\t"
            "global_load_dwordx4 %6, %8, off offset:2048 sc0 sc1\n\t"
            "global_load_dwordx4 %7, %8, off offset:3072 sc0 sc1\n\t"
            "s_waitcnt vmcnt(0)"
            : "=&v"(Af[0]), "=&v"(Af[1]), "=&v"(Af[2]), "=&v"(Af[3]), "=&v"(Af[4]), "=&v"(Af[5]), "=&v"(Af[6]), "=&v"(Af[7])
            : "v"(a) : "memory");
        gemm_block(p, sm, Af, Bf, (int)srcr * MS + 16 * b, col0, scale, bad);
        STAMP(4 + 2 * t);
    }
    (void)PER;
}

__global__ void __launch_bounds__(NTHREADS, 2) ag_gemm_kernel(Params p) {
    __shared__ Smem sm;
    if (blockIdx.x < NPUSH_WG) pusher(p);
    else gemm_wg(p, sm);
}

}

extern "C" void kernel_launch(int rank, int nranks, ncclComm_t comm, hipStream_t stream,
                              void* const* d_in, const int* in_sizes, int n_in,
                              void* d_out, int out_size, const int* in_dtypes, int out_dtype,
                              void* const* d_out_all, void* const* d_ws_all, size_t ws_bytes) {
    (void)comm; (void)in_dtypes; (void)out_dtype; (void)d_out_all;
    if (nranks != NR || n_in != 4 || (unsigned)rank >= (unsigned)NR || ws_bytes < WS_NEED ||
        in_sizes[0] != MS * KD || in_sizes[1] != KD * NS || out_size != NR * MS * NS) {
        fprintf(stderr, "kernel_launch: unexpected problem shape\n");
        return;
    }
    Params p;
    p.x = (const uint8_t*)d_in[0];
    p.w = (const uint8_t*)d_in[1];
    p.sx = (const float*)d_in[2];
    p.sw = (const float*)d_in[3];
    p.out = (float*)d_out;
    for (int i = 0; i < NR; ++i) p.ws[i] = (uint8_t*)d_ws_all[i];
    p.rank = rank;
    p.pad = 0;
    hipLaunchKernelGGL(ag_gemm_kernel, dim3(NGRID), dim3(NTHREADS), 0, stream, p);
    hipError_t e = hipGetLastError();
    if (e != hipSuccess) fprintf(stderr, "launch error %s\n", hipGetErrorString(e));
}
```

```cpp
#include <hip/hip_runtime.h>
#include <rccl/rccl.h>
#include <cstdio>
#include <cstdint>

typedef unsigned u32x4 __attribute__((ext_vector_type(4)));
typedef int i32x8 __attribute__((ext_vector_type(8)));
typedef float f32x4 __attribute__((ext_vector_type(4)));
typedef float f32x2 __attribute__((ext_vector_type(2)));

namespace {

constexpr int NR = 8;
constexpr int KD = 4096;
constexpr int MS = 512;
constexpr int NS = 512;
constexpr int NBLK = 32;
constexpr int BLK_BYTES = 16 * KD;
constexpr int SHARD_BYTES = MS * KD;
constexpr int NGRID = 256;
constexpr int NTHREADS = 512;
constexpr int NGROUP = 24;

constexpr size_t CNT_OFF = 0;
constexpr size_t TMO_OFF = 32768;
constexpr size_t FLAG_OFF = 65536;
constexpr size_t DBG_OFF = 131072;
constexpr size_t STAGE_OFF = 1u << 20;
constexpr size_t SLOT_BYTES = (size_t)NR * SHARD_BYTES;
constexpr size_t WS_NEED = STAGE_OFF + 2 * SLOT_BYTES;

constexpr unsigned long long DEADLINE_TICKS = 40000000ull;

struct Params {
    const uint8_t* x;
    const uint8_t* w;
    const float* sx;
    const float* sw;
    float* out;
    uint8_t* me;
    uint8_t* peer[7];
    int rank;
    int pad;
};

__device__ __forceinline__ unsigned ld_sys(const unsigned* p) {
    return __hip_atomic_load(p, __ATOMIC_RELAXED, __HIP_MEMORY_SCOPE_SYSTEM);
}
__device__ __forceinline__ unsigned ld_agent(const unsigned* p) {
    return __hip_atomic_load(p, __ATOMIC_RELAXED, __HIP_MEMORY_SCOPE_AGENT);
}

#define STAMP(idx) do {} while (0)

#define LDBITS "sc0 sc1"
#define WAITV(n) asm volatile("s_waitcnt vmcnt(" #n ")" ::: "memory")

__device__ __forceinline__ unsigned draw_call(const Params& p) {
    unsigned* cnt = (unsigned*)(p.me + CNT_OFF + (size_t)blockIdx.x * 128) + (threadIdx.x >> 6);
    unsigned c = 0;
    if ((threadIdx.x & 63) == 0) c = __hip_atomic_fetch_add(cnt, 1u, __ATOMIC_RELAXED, __HIP_MEMORY_SCOPE_AGENT) + 1u;
    return (unsigned)__builtin_amdgcn_readfirstlane((int)c);
}

__device__ __forceinline__ const uint8_t* uniform_ptr(const uint8_t* q) {
    const unsigned long long v = (unsigned long long)q;
    const unsigned lo = (unsigned)__builtin_amdgcn_readfirstlane((int)(unsigned)v);
    const unsigned hi = (unsigned)__builtin_amdgcn_readfirstlane((int)(unsigned)(v >> 32));
    return (const uint8_t*)(((unsigned long long)hi << 32) | lo);
}

struct alignas(16) Smem {
    f32x4 red[2][8][16][16];
    unsigned cnt[NBLK];
    unsigned bad[2];
};

#define LOAD2(R, ptr) asm volatile( \
    "global_load_dwordx4 %0, %2, off\n\t" \
    "global_load_dwordx4 %1, %2, off offset:64" \
    : "=&v"(R[0]), "=&v"(R[1]) : "v"(ptr) : "memory")
#define STORE2(R, ptr) asm volatile( \
    "global_store_dwordx4 %2, %0, off sc0 sc1\n\t" \
    "global_store_dwordx4 %2, %1, off offset:1024 sc0 sc1\n\t" \
    "s_nop 1" \
    :: "v"(R[0]), "v"(R[1]), "v"(ptr) : "memory")

__device__ __forceinline__ void pusher(const Params& p, Smem& sm, int dd, int w) {
    const int tid = threadIdx.x;
    if (tid < NBLK) sm.cnt[tid] = 0u;
    __syncthreads();
    const int s = __builtin_amdgcn_readfirstlane(tid >> 6);
    if (s > 4) return;
    const int l = tid & 63, r = l & 15, q = l >> 4;
    uint8_t* pw = p.peer[dd];
    if (s == 4) {
        unsigned* flag = (unsigned*)(pw + FLAG_OFF + (size_t)(p.rank * 8 + w) * 128);
        const unsigned call = draw_call(p);
        const unsigned base = (call - 1u) * NBLK;
        const unsigned long long t0 = __builtin_amdgcn_s_memrealtime();
#pragma unroll 1
        for (int t = 0; t < NBLK; ++t) {
            unsigned itn = 0;
            while (__hip_atomic_load(&sm.cnt[t], __ATOMIC_RELAXED, __HIP_MEMORY_SCOPE_WORKGROUP) < 4u) {
                if ((++itn & 1023u) == 0u && __builtin_amdgcn_s_memrealtime() - t0 > DEADLINE_TICKS) return;
                __builtin_amdgcn_s_sleep(1);
            }
            if (l == 0) __hip_atomic_fetch_max(flag, base + t + 1u, __ATOMIC_RELAXED, __HIP_MEMORY_SCOPE_SYSTEM);
        }
        return;
    }
    STAMP(0);
    const uint8_t* src = p.x + (size_t)r * KD + 512 * w + 128 * s + 16 * q;
    u32x4 A[2], B[2];
    LOAD2(A, src);
    LOAD2(B, src + BLK_BYTES);
    const unsigned call = draw_call(p);
    uint8_t* dst = pw + STAGE_OFF + (size_t)(call & 1u) * SLOT_BYTES + (size_t)p.rank * SHARD_BYTES + w * 8192 + s * 2048 + l * 16;
    WAITV(0);
    STAMP(1);
#define SIGNAL(t) do { if (l == 0) __hip_atomic_fetch_add(&sm.cnt[t], 1u, __ATOMIC_RELAXED, __HIP_MEMORY_SCOPE_WORKGROUP); } while (0)
#pragma unroll 1
    for (int t = 0; t < NBLK - 2; t += 2) {
        if (t > 0) { WAITV(4); SIGNAL(t - 2); }
        STORE2(A, dst + (size_t)t * BLK_BYTES);
        LOAD2(A, src + (size_t)(t + 2) * BLK_BYTES);
        if (t > 0) { WAITV(4); SIGNAL(t - 1); }
        STORE2(B, dst + (size_t)(t + 1) * BLK_BYTES);
        LOAD2(B, src + (size_t)(t + 3) * BLK_BYTES);
        if (t == 14) STAMP(2);
    }
    WAITV(4); SIGNAL(NBLK - 4);
    STORE2(A, dst + (size_t)(NBLK - 2) * BLK_BYTES);
    WAITV(2); SIGNAL(NBLK - 3);
    STORE2(B, dst + (size_t)(NBLK - 1) * BLK_BYTES);
    WAITV(2); SIGNAL(NBLK - 2);
    WAITV(0); SIGNAL(NBLK - 1);
    STAMP(3);
#undef SIGNAL
}

__device__ __forceinline__ void transpose4(unsigned a, unsigned b, unsigned c, unsigned d,
                                           unsigned& o0, unsigned& o1, unsigned& o2, unsigned& o3) {
    const unsigned t0 = __builtin_amdgcn_perm(b, a, 0x05010400u);
    const unsigned t1 = __builtin_amdgcn_perm(b, a, 0x07030602u);
    const unsigned t2 = __builtin_amdgcn_perm(d, c, 0x05010400u);
    const unsigned t3 = __builtin_amdgcn_perm(d, c, 0x07030602u);
    o0 = __builtin_amdgcn_perm(t2, t0, 0x05040100u);
    o1 = __builtin_amdgcn_perm(t2, t0, 0x07060302u);
    o2 = __builtin_amdgcn_perm(t3, t1, 0x05040100u);
    o3 = __builtin_amdgcn_perm(t3, t1, 0x07060302u);
}

__device__ __forceinline__ i32x8 join8(u32x4 lo, u32x4 hi) {
    i32x8 v;
    v[0] = lo[0]; v[1] = lo[1]; v[2] = lo[2]; v[3] = lo[3];
    v[4] = hi[0]; v[5] = hi[1]; v[6] = hi[2]; v[7] = hi[3];
    return v;
}

__device__ __forceinline__ void gemm_wg(const Params& p, Smem& sm, int g, int j) {
    const int tid = threadIdx.x, w = __builtin_amdgcn_readfirstlane(tid >> 6), l = tid & 63, r = l & 15, q = l >> 4;
    STAMP(0);
    if (tid < 2) sm.bad[tid] = 0u;
#define WDELAY 8
    for (int i = 0; i < WDELAY; ++i) __builtin_amdgcn_s_sleep(2);
    i32x8 Bf[4][4];
    {
        const uint8_t* wp = p.w + (size_t)(512 * w + 16 * q) * NS + 64 * j + 4 * r;
#pragma unroll
        for (int s = 0; s < 4; ++s) {
            unsigned t[32];
#pragma unroll
            for (int h = 0; h < 2; ++h)
#pragma unroll
                for (int k = 0; k < 16; ++k)
                    t[h * 16 + k] = *(const unsigned*)(wp + (size_t)(128 * s + 64 * h + k) * NS);
#pragma unroll
            for (int jj = 0; jj < 8; ++jj) {
                unsigned o0, o1, o2, o3;
                transpose4(t[4 * jj], t[4 * jj + 1], t[4 * jj + 2], t[4 * jj + 3], o0, o1, o2, o3);
                Bf[s][0][jj] = o0; Bf[s][1][jj] = o1; Bf[s][2][jj] = o2; Bf[s][3][jj] = o3;
            }
        }
    }
    const float scale = p.sx[0] * p.sw[0];
    const unsigned call = draw_call(p);
    __syncthreads();
    STAMP(1);
    unsigned* tmo = (unsigned*)(p.me + TMO_OFF);
    const uint8_t* stage = p.me + STAGE_OFF + (size_t)(call & 1u) * SLOT_BYTES + w * 8192;
    const uint8_t* xloc = p.x + 512 * w;
    float* outp = p.out + (size_t)(tid >> 5) * NS + 64 * j + (tid & 31) * 2;
    const int nloc = g >= 16 ? 2 : 1;
    const int nitems = nloc + (g < 8 ? 10 : 9);
    bool bad = false;
    for (int it = 0; it < nitems; ++it) {
        const bool local = it < nloc;
        const uint8_t* base;
        unsigned voff, stride;
        int row0;
        if (local) {
            const int lb = g + 8 * it;
            base = xloc + (size_t)lb * BLK_BYTES;
            voff = r * KD + 16 * q;
            stride = 64;
            row0 = p.rank * MS + 16 * lb;
        } else {
            const int n = g + NGROUP * (it - nloc);
            const int i = n % 7, b = n / 7;
            const unsigned srcr = (unsigned)(p.rank + 1 + i) & 7u;
            base = stage + (size_t)srcr * SHARD_BYTES + (size_t)b * BLK_BYTES;
            voff = l * 16;
            stride = 1024;
            row0 = (int)srcr * MS + 16 * b;
            if (!bad) {
                const unsigned* flag = (const unsigned*)(p.me + FLAG_OFF + (size_t)(srcr * 8 + w) * 128);
                const unsigned seq = (call - 1u) * NBLK + b + 1u;
                if (ld_sys(flag) < seq) {
                    const unsigned long long t0 = __builtin_amdgcn_s_memrealtime();
                    unsigned itn = 0;
                    while (ld_sys(flag) < seq) {
                        if ((++itn & 15u) == 0u &&
                            (ld_agent(tmo) == call || __builtin_amdgcn_s_memrealtime() - t0 > DEADLINE_TICKS)) {
                            if (l == 0) __hip_atomic_store(tmo, call, __ATOMIC_RELAXED, __HIP_MEMORY_SCOPE_AGENT);
                            bad = true;
                            break;
                        }
                        __builtin_amdgcn_s_sleep(1);
                    }
                }
            }
        }
        STAMP(2 + 2 * it);
        base = uniform_ptr(base);
        u32x4 Af[8];
        {
            const unsigned v1 = voff + stride, v2 = v1 + stride, v3 = v2 + stride, v4 = v3 + stride, v5 = v4 + stride,
                           v6 = v5 + stride, v7 = v6 + stride;
            asm volatile(
                "global_load_dwordx4 %0, %8, %16 " LDBITS "\n\t"
                "global_load_dwordx4 %1, %9, %16 " LDBITS "\n\t"
                "global_load_dwordx4 %2, %10, %16 " LDBITS "\n\t"
                "global_load_dwordx4 %3, %11, %16 " LDBITS "\n\t"
                "global_load_dwordx4 %4, %12, %16 " LDBITS "\n\t"
                "global_load_dwordx4 %5, %13, %16 " LDBITS "\n\t"
                "global_load_dwordx4 %6, %14, %16 " LDBITS "\n\t"
                "global_load_dwordx4 %7, %15, %16 " LDBITS
                : "=&v"(Af[0]), "=&v"(Af[1]), "=&v"(Af[2]), "=&v"(Af[3]), "=&v"(Af[4]), "=&v"(Af[5]), "=&v"(Af[6]), "=&v"(Af[7])
                : "v"(voff), "v"(v1), "v"(v2), "v"(v3), "v"(v4), "v"(v5), "v"(v6), "v"(v7), "s"(base) : "memory");
        }
        f32x4 acc[4];
#pragma unroll
        for (int i = 0; i < 4; ++i) acc[i] = f32x4{0.f, 0.f, 0.f, 0.f};
#define KSTEP(s, n) { \
            asm volatile("s_waitcnt vmcnt(" #n ")" : "+v"(Af[2 * s]), "+v"(Af[2 * s + 1]) :: "memory"); \
            const i32x8 a = join8(Af[2 * s], Af[2 * s + 1]); \
            _Pragma("unroll") for (int i = 0; i < 4; ++i) \
                acc[i] = __builtin_amdgcn_mfma_scale_f32_16x16x128_f8f6f4(a, Bf[s][i], acc[i], 0, 0, 0, 0, 0, 0); \
            __builtin_amdgcn_sched_barrier(0); }
        KSTEP(0, 6) KSTEP(1, 4) KSTEP(2, 2) KSTEP(3, 0)
#undef KSTEP
        const int buf = it & 1;
#pragma unroll
        for (int t = 0; t < 4; ++t) {
            f32x4 v = f32x4{acc[0][t], acc[1][t], acc[2][t], acc[3][t]};
            asm volatile("" : "+v"(v));
            sm.red[buf][w][4 * q + t][r] = v;
        }
        if (bad) sm.bad[buf] = 1u;
        __syncthreads();
        const bool anybad = sm.bad[buf] != 0u;
        const f32x2* rp = (const f32x2*)&sm.red[buf][0][0][0] + tid;
        f32x2 v = rp[0];
#pragma unroll
        for (int k = 1; k < 8; ++k) v += rp[k * 512];
        v *= scale;
        v[0] = fmaxf(v[0], 0.f);
        v[1] = fmaxf(v[1], 0.f);
        if (anybad) v = f32x2{__builtin_nanf(""), __builtin_nanf("")};
        *(f32x2*)(outp + (size_t)row0 * NS) = v;
        STAMP(3 + 2 * it);
    }
}

__global__ void __launch_bounds__(NTHREADS, 2) ag_gemm_kernel(Params p) {
    __shared__ Smem sm;
    const int xcd = blockIdx.x & 7, slot = blockIdx.x >> 3;
    if (slot < 7) pusher(p, sm, slot, xcd);
    else if (slot >= 8) gemm_wg(p, sm, 3 * xcd + ((slot - 8) >> 3), (slot - 8) & 7);
}

}

extern "C" void kernel_launch(int rank, int nranks, ncclComm_t comm, hipStream_t stream,
                              void* const* d_in, const int* in_sizes, int n_in,
                              void* d_out, int out_size, const int* in_dtypes, int out_dtype,
                              void* const* d_out_all, void* const* d_ws_all, size_t ws_bytes) {
    (void)comm; (void)in_dtypes; (void)out_dtype; (void)d_out_all;
    if (nranks != NR || n_in != 4 || (unsigned)rank >= (unsigned)NR || ws_bytes < WS_NEED ||
        in_sizes[0] != MS * KD || in_sizes[1] != KD * NS || out_size != NR * MS * NS) {
        fprintf(stderr, "kernel_launch: unexpected problem shape\n");
        return;
    }
    Params p;
    p.x = (const uint8_t*)d_in[0];
    p.w = (const uint8_t*)d_in[1];
    p.sx = (const float*)d_in[2];
    p.sw = (const float*)d_in[3];
    p.out = (float*)d_out;
    p.me = (uint8_t*)d_ws_all[rank];
    for (int i = 0; i < 7; ++i) p.peer[i] = (uint8_t*)d_ws_all[(rank + 1 + i) % NR];
    p.rank = rank;
    p.pad = 0;
    hipLaunchKernelGGL(ag_gemm_kernel, dim3(NGRID), dim3(NTHREADS), 0, stream, p);
    hipError_t e = hipGetLastError();
    if (e != hipSuccess) fprintf(stderr, "launch error %s\n", hipGetErrorString(e));
}
```

```cpp
#include <hip/hip_runtime.h>
#include <rccl/rccl.h>
#include <cstdio>
#include <cstdint>

typedef unsigned u32x4 __attribute__((ext_vector_type(4)));
typedef unsigned u32x3 __attribute__((ext_vector_type(3)));
typedef int i32x8 __attribute__((ext_vector_type(8)));
typedef float f32x4 __attribute__((ext_vector_type(4)));
typedef float f32x2 __attribute__((ext_vector_type(2)));

namespace {

constexpr int NR = 8;
constexpr int KD = 4096;
constexpr int MS = 512;
constexpr int NS = 512;
constexpr int NBLK = 32;
constexpr int BLK_BYTES = 16 * KD;
constexpr int NGRID = 256;
constexpr int NTHREADS = 512;
constexpr int NGROUP = 24;

constexpr int HI_OFF = 4096, LIST_OFF = 7168, RAW7_OFF = 7424;
constexpr int PIECE_BYTES = 8448;
constexpr int SBLK_BYTES = 8 * PIECE_BYTES;
constexpr int SSHARD_BYTES = NBLK * SBLK_BYTES;
constexpr unsigned LIST_CAP = 124;
constexpr int ABUF_BYTES = 7424;
constexpr int MAXITEMS = 12;

constexpr size_t CNT_OFF = 0;
constexpr size_t TMO_OFF = 32768;
constexpr size_t FLAG_OFF = 65536;
constexpr size_t DBG_OFF = 131072;
constexpr size_t STAGE_OFF = 1u << 20;
constexpr size_t SLOT_BYTES = (size_t)NR * SSHARD_BYTES;
constexpr size_t WS_NEED = STAGE_OFF + 2 * SLOT_BYTES;

constexpr unsigned long long DEADLINE_TICKS = 40000000ull;

struct Params {
    const uint8_t* x;
    const uint8_t* w;
    const float* sx;
    const float* sw;
    float* out;
    uint8_t* me;
    uint8_t* peer[7];
    int rank;
    int pad;
};

__device__ __forceinline__ unsigned ld_sys(const unsigned* p) {
    return __hip_atomic_load(p, __ATOMIC_RELAXED, __HIP_MEMORY_SCOPE_SYSTEM);
}
__device__ __forceinline__ unsigned ld_agent(const unsigned* p) {
    return __hip_atomic_load(p, __ATOMIC_RELAXED, __HIP_MEMORY_SCOPE_AGENT);
}

#define STAMP(idx) do {} while (0)
#define STAMP_INIT() do {} while (0)
#define STAMP_FLUSH() do {} while (0)

#define DSTAMP(idx) do {} while (0)
#define WSTAMP(idx) do {} while (0)
#define WAITV(n) asm volatile("s_waitcnt vmcnt(" #n ")" ::: "memory")

__device__ __forceinline__ unsigned draw_call(const Params& p) {
    unsigned* cnt = (unsigned*)(p.me + CNT_OFF + (size_t)blockIdx.x * 128) + (threadIdx.x >> 6);
    unsigned c = 0;
    if ((threadIdx.x & 63) == 0) c = __hip_atomic_fetch_add(cnt, 1u, __ATOMIC_RELAXED, __HIP_MEMORY_SCOPE_AGENT) + 1u;
    return (unsigned)__builtin_amdgcn_readfirstlane((int)c);
}

__device__ __forceinline__ const uint8_t* uniform_ptr(const uint8_t* q) {
    const unsigned long long v = (unsigned long long)q;
    const unsigned lo = (unsigned)__builtin_amdgcn_readfirstlane((int)(unsigned)v);
    const unsigned hi = (unsigned)__builtin_amdgcn_readfirstlane((int)(unsigned)(v >> 32));
    return (const uint8_t*)(((unsigned long long)hi << 32) | lo);
}

struct alignas(16) Smem {
    union {
        struct {
            f32x4 red[2][8][16][16];
            u32x4 abuf[8][ABUF_BYTES / 16];
            unsigned mask[8][256];
            unsigned poll[8][16];
            unsigned bad[2];
        } g;
        struct {
            u32x4 ring[4][4][128];
            unsigned list[NBLK][64];
            unsigned alloc[NBLK];
            unsigned valid[NBLK];
            unsigned rawb[NBLK];
            unsigned packed[NBLK];
            unsigned done[NBLK];
        } s;
    };
};

__device__ __forceinline__ unsigned and_or(unsigned a, unsigned m, unsigned c) { return (a & m) | c; }
__device__ __forceinline__ unsigned bfi(unsigned m, unsigned a, unsigned b) { return (a & m) | (b & ~m); }

__device__ __forceinline__ unsigned wave_max(unsigned v) {
    v = max(v, (unsigned)__builtin_amdgcn_update_dpp(0, (int)v, 0x111, 0xf, 0xf, true));
    v = max(v, (unsigned)__builtin_amdgcn_update_dpp(0, (int)v, 0x112, 0xf, 0xf, true));
    v = max(v, (unsigned)__builtin_amdgcn_update_dpp(0, (int)v, 0x114, 0xf, 0xf, true));
    v = max(v, (unsigned)__builtin_amdgcn_update_dpp(0, (int)v, 0x118, 0xf, 0xf, true));
    v = max(v, (unsigned)__builtin_amdgcn_update_dpp(0, (int)v, 0x142, 0xa, 0xf, true));
    v = max(v, (unsigned)__builtin_amdgcn_update_dpp(0, (int)v, 0x143, 0xc, 0xf, true));
    return (unsigned)__builtin_amdgcn_readlane((int)v, 63);
}
__device__ __forceinline__ unsigned wave_sum(unsigned v) {
    v += (unsigned)__builtin_amdgcn_update_dpp(0, (int)v, 0x111, 0xf, 0xf, true);
    v += (unsigned)__builtin_amdgcn_update_dpp(0, (int)v, 0x112, 0xf, 0xf, true);
    v += (unsigned)__builtin_amdgcn_update_dpp(0, (int)v, 0x114, 0xf, 0xf, true);
    v += (unsigned)__builtin_amdgcn_update_dpp(0, (int)v, 0x118, 0xf, 0xf, true);
    v += (unsigned)__builtin_amdgcn_update_dpp(0, (int)v, 0x142, 0xa, 0xf, true);
    v += (unsigned)__builtin_amdgcn_update_dpp(0, (int)v, 0x143, 0xc, 0xf, true);
    return (unsigned)__builtin_amdgcn_readlane((int)v, 63);
}

__device__ __forceinline__ void fetch_unit(const uint8_t* src, unsigned lds) {
    unsigned keep;
    asm volatile(
        "s_mov_b32 %0, m0\n\t"
        "s_mov_b32 m0, %3\n\t"
        "s_nop 0\n\t"
        "global_load_lds_dwordx4 %1, off\n\t"
        "s_add_u32 m0, %3, 1024\n\t"
        "s_nop 0\n\t"
        "global_load_lds_dwordx4 %2, off\n\t"
        "s_mov_b32 m0, %0"
        : "=&s"(keep) : "v"(src), "v"(src + 64), "s"(lds) : "memory", "scc");
}

struct Packed {
    u32x4 lo;
    u32x3 hi;
    unsigned d7;
    bool raw;
};

__device__ __forceinline__ void pack_unit(const u32x4* slot, Smem& sm, int t, int s, int l, Packed& o) {
    const u32x4 R0 = slot[l], R1 = slot[64 + l];
    const unsigned f[8] = {R0[0], R0[1], R0[2], R0[3], R1[0], R1[1], R1[2], R1[3]};
    const unsigned e0 = max(f[0] & 0x7Fu, f[4] & 0x7Fu), e1 = max((f[0] >> 8) & 0x7Fu, (f[4] >> 8) & 0x7Fu);
    const unsigned e2 = max((f[0] >> 16) & 0x7Fu, (f[4] >> 16) & 0x7Fu), e3 = max((f[0] >> 24) & 0x7Fu, (f[4] >> 24) & 0x7Fu);
    const unsigned mx = wave_max(max(max(e0, e1), max(e2, e3)));
    const unsigned a = mx > 63u ? mx - 63u : 0u;
    const unsigned na = (128u - a) * 0x01010101u;
    unsigned tt[8], m = 0;
#pragma unroll
    for (int j = 0; j < 8; ++j) tt[j] = (f[j] & 0x7F7F7F7Fu) + na;
#pragma unroll
    for (int j = 0; j < 7; ++j) m = and_or(tt[j] >> (6 - j), 0x01010101u << j, m);
    m = and_or(tt[7] << 1, 0x80808080u, m);
    const unsigned n = wave_sum((unsigned)__popc(m));
    unsigned base = LIST_CAP + 1u;
    if (n <= LIST_CAP) {
        unsigned bb = 0;
        if (l == 0) bb = __hip_atomic_fetch_add(&sm.s.alloc[t], n, __ATOMIC_RELAXED, __HIP_MEMORY_SCOPE_WORKGROUP);
        base = (unsigned)__builtin_amdgcn_readfirstlane((int)bb);
    }
    o.raw = base + n > LIST_CAP;
    if (!o.raw) {
        unsigned short* l16 = (unsigned short*)sm.s.list[t] + 4;
        unsigned pos = base, mm = m;
        const unsigned tag = (unsigned)((s << 11) | (l << 5));
        for (;;) {
            const unsigned long long bal = __ballot(mm != 0u);
            if (bal == 0ull) break;
            if (mm != 0u) {
                const unsigned idx = pos + __builtin_amdgcn_mbcnt_hi((unsigned)(bal >> 32), __builtin_amdgcn_mbcnt_lo((unsigned)bal, 0u));
                l16[idx] = (unsigned short)(tag | (unsigned)(__ffs(mm) - 1));
                mm &= mm - 1u;
            }
            pos += (unsigned)__popcll(bal);
        }
        if (l == 0) __hip_atomic_fetch_max(&sm.s.valid[t], base + n, __ATOMIC_RELAXED, __HIP_MEMORY_SCOPE_WORKGROUP);
    } else if (l == 0) {
        __hip_atomic_fetch_or(&sm.s.rawb[t], 1u << s, __ATOMIC_RELAXED, __HIP_MEMORY_SCOPE_WORKGROUP);
    }
    if (l == 0) {
        ((unsigned char*)&sm.s.list[t][1])[s] = (unsigned char)a;
        asm volatile("s_waitcnt lgkmcnt(0)" ::: "memory");
        __hip_atomic_fetch_add(&sm.s.packed[t], 1u, __ATOMIC_RELAXED, __HIP_MEMORY_SCOPE_WORKGROUP);
    }
    const unsigned p7 = and_or(tt[7], 0x3F3F3F3Fu, f[7] & 0x80808080u);
    unsigned d[7];
#pragma unroll
    for (int j = 0; j < 7; ++j) {
        const unsigned sp = j < 6 ? (p7 << (6 - j)) : (p7 >> 1);
        d[j] = and_or(sp, 0x40404040u, and_or(tt[j], 0x3F3F3F3Fu, f[j] & 0x80808080u));
    }
    if (o.raw) {
        o.lo = R0;
        o.hi = u32x3{f[4], f[5], f[6]};
    } else {
        o.lo = u32x4{d[0], d[1], d[2], d[3]};
        o.hi = u32x3{d[4], d[5], d[6]};
    }
    o.d7 = f[7];
}

__device__ __forceinline__ void store_unit(const Packed& o, uint8_t* piece, int s, int l) {
    asm volatile(
        "global_store_dwordx4 %0, %1, %4 sc0 sc1\n\t"
        "global_store_dwordx3 %2, %3, %4 sc0 sc1\n\t"
        "s_nop 1"
        :: "v"(s * 1024 + l * 16), "v"(o.lo), "v"(HI_OFF + s * 768 + l * 12), "v"(o.hi), "s"(piece) : "memory");
    if (o.raw)
        asm volatile("global_store_dword %0, %1, %2 sc0 sc1\n\ts_nop 1"
                     :: "v"(RAW7_OFF + s * 256 + l * 4), "v"(o.d7), "s"(piece) : "memory");
}

__device__ __forceinline__ bool lds_wait(const unsigned* word, unsigned value, unsigned long long t0) {
    unsigned itn = 0;
    while (__hip_atomic_load(word, __ATOMIC_RELAXED, __HIP_MEMORY_SCOPE_WORKGROUP) < value) {
        if ((++itn & 1023u) == 0u && __builtin_amdgcn_s_memrealtime() - t0 > DEADLINE_TICKS) return false;
        __builtin_amdgcn_s_sleep(1);
    }
    asm volatile("" ::: "memory");
    return true;
}

__device__ __forceinline__ void pusher(const Params& p, Smem& sm, int dd, int w) {
    const int tid = threadIdx.x;
    if (tid < NBLK) { sm.s.alloc[tid] = 0u; sm.s.valid[tid] = 0u; sm.s.rawb[tid] = 0u; sm.s.packed[tid] = 0u; sm.s.done[tid] = 0u; }
    __syncthreads();
    const int s = __builtin_amdgcn_readfirstlane(tid >> 6);
    if (s > 4) return;
    const int l = tid & 63, r = l & 15, q = l >> 4;
    uint8_t* pw = p.peer[dd];
    if (s == 4) {
        unsigned* flag = (unsigned*)(pw + FLAG_OFF + (size_t)(p.rank * 8 + w) * 128);
        const unsigned call = draw_call(p);
        const unsigned base = (call - 1u) * NBLK;
        uint8_t* dst = (uint8_t*)uniform_ptr(pw + STAGE_OFF + (size_t)(call & 1u) * SLOT_BYTES + (size_t)p.rank * SSHARD_BYTES +
                                             w * PIECE_BYTES + LIST_OFF);
        const unsigned long long t0 = __builtin_amdgcn_s_memrealtime();
#define PUBLISH(t) do { unsigned long long sv_; \
            asm volatile("s_mov_b64 %0, exec\n\ts_mov_b64 exec, 1\n\tglobal_atomic_umax %1, %2, off sc1\n\ts_mov_b64 exec, %0" \
                         : "=&s"(sv_) : "v"(flag), "v"(base + (unsigned)(t) + 1u) : "memory"); } while (0)
#pragma unroll 1
        for (int t = 0; t < NBLK; ++t) {
            if (!lds_wait(&sm.s.packed[t], 4u, t0)) return;
            const unsigned nv = sm.s.valid[t];
            unsigned v = sm.s.list[t][l];
            if (l == 0) v = nv | (sm.s.rawb[t] << 16);
            const unsigned nl = 2u + (nv + 1u) / 2u;
            const unsigned long long em = nl >= 64u ? ~0ull : ((1ull << nl) - 1ull);
            unsigned long long sv;
            asm volatile("s_mov_b64 %0, exec\n\ts_mov_b64 exec, %4\n\tglobal_store_dword %1, %2, %3 sc0 sc1\n\ts_mov_b64 exec, %0"
                         : "=&s"(sv) : "v"(l * 4), "v"(v), "s"(dst + (size_t)t * SBLK_BYTES), "s"(em) : "memory");
            if (t >= 2) {
                if (!lds_wait(&sm.s.done[t - 2], 4u, t0)) return;
                if (t >= 4) WAITV(4); else WAITV(2);
                PUBLISH(t - 2);
            }
        }
        if (!lds_wait(&sm.s.done[NBLK - 2], 4u, t0)) return;
        WAITV(3);
        PUBLISH(NBLK - 2);
        if (!lds_wait(&sm.s.done[NBLK - 1], 4u, t0)) return;
        WAITV(2);
        PUBLISH(NBLK - 1);
#undef PUBLISH
        return;
    }
    STAMP_INIT();
    STAMP(0);
    const uint8_t* src = p.x + (size_t)r * KD + 512 * w + 128 * s + 16 * q;
    u32x4* ring = &sm.s.ring[s][0][0];
    const unsigned ring_lds = (unsigned)__builtin_amdgcn_readfirstlane((int)(unsigned)(uintptr_t)ring);
#define WIN 2
#define PACE 1648
    fetch_unit(src, ring_lds);
    fetch_unit(src + BLK_BYTES, ring_lds + 2048u);
    if (WIN == 3) fetch_unit(src + 2 * BLK_BYTES, ring_lds + 4096u);
    const unsigned call = draw_call(p);
    uint8_t* dst = (uint8_t*)uniform_ptr(pw + STAGE_OFF + (size_t)(call & 1u) * SLOT_BYTES + (size_t)p.rank * SSHARD_BYTES +
                                         w * PIECE_BYTES);
    WAITV(0);
    STAMP(1);
    const unsigned long long tstart = __builtin_amdgcn_s_memrealtime();
#define SIGNAL(t) do { if (l == 0) __hip_atomic_fetch_add(&sm.s.done[t], 1u, __ATOMIC_RELAXED, __HIP_MEMORY_SCOPE_WORKGROUP); } while (0)
#pragma unroll 1
    for (int t = 0; t < NBLK; ++t) {
        if (WIN == 2) { if (t >= 2) WAITV(6); }
        else { if (t >= 3) WAITV(10); }
        Packed o;
        pack_unit(ring + (t & 3) * 128, sm, t, s, l, o);
        const int tn = t + WIN < NBLK ? t + WIN : t;
        fetch_unit(src + (size_t)tn * BLK_BYTES, ring_lds + (unsigned)((t + WIN) & 3) * 2048u);
        if (WIN == 2) { if (t >= 2) { WAITV(6); SIGNAL(t - 2); } }
        else { if (t >= 3) { WAITV(10); SIGNAL(t - 3); } }
        if (PACE > 0) {
            const unsigned long long due = tstart + (((unsigned long long)t * PACE) >> 4);
            while (__builtin_amdgcn_s_memrealtime() < due) __builtin_amdgcn_s_sleep(1);
        }
        store_unit(o, dst + (size_t)t * SBLK_BYTES, s, l);
        if (t == 15) STAMP(2);
    }
    if (WIN == 3) { WAITV(8); SIGNAL(NBLK - 3); }
    WAITV(4); SIGNAL(NBLK - 2);
    WAITV(0); SIGNAL(NBLK - 1);
    STAMP(3);
    STAMP_FLUSH();
#undef SIGNAL
}

__device__ __forceinline__ void transpose4(unsigned a, unsigned b, unsigned c, unsigned d,
                                           unsigned& o0, unsigned& o1, unsigned& o2, unsigned& o3) {
    const unsigned t0 = __builtin_amdgcn_perm(b, a, 0x05010400u);
    const unsigned t1 = __builtin_amdgcn_perm(b, a, 0x07030602u);
    const unsigned t2 = __builtin_amdgcn_perm(d, c, 0x05010400u);
    const unsigned t3 = __builtin_amdgcn_perm(d, c, 0x07030602u);
    o0 = __builtin_amdgcn_perm(t2, t0, 0x05040100u);
    o1 = __builtin_amdgcn_perm(t2, t0, 0x07060302u);
    o2 = __builtin_amdgcn_perm(t3, t1, 0x05040100u);
    o3 = __builtin_amdgcn_perm(t3, t1, 0x07060302u);
}

__device__ __forceinline__ i32x8 join8(u32x4 lo, u32x4 hi) {
    i32x8 v;
    v[0] = lo[0]; v[1] = lo[1]; v[2] = lo[2]; v[3] = lo[3];
    v[4] = hi[0]; v[5] = hi[1]; v[6] = hi[2]; v[7] = hi[3];
    return v;
}

__device__ __forceinline__ i32x8 unpack_unit(u32x4 lo, u32x3 hi, unsigned m, unsigned A) {
    const unsigned d[7] = {lo[0], lo[1], lo[2], lo[3], hi[0], hi[1], hi[2]};
    i32x8 f;
    unsigned g = (d[6] << 1) & 0x80808080u;
#pragma unroll
    for (int j = 0; j < 6; ++j) g = and_or(d[j] >> (6 - j), 0x01010101u << j, g);
#pragma unroll
    for (int j = 0; j < 7; ++j) f[j] = (int)(((d[j] & 0xBFBFBFBFu) + A) ^ ((m << (6 - j)) & 0x40404040u));
    f[7] = (int)((g + A) ^ ((m >> 1) & 0x40404040u));
    return f;
}

__device__ __forceinline__ void gemm_wg(const Params& p, Smem& sm, int g, int j) {
    const int tid = threadIdx.x, w = __builtin_amdgcn_readfirstlane(tid >> 6), l = tid & 63, r = l & 15, q = l >> 4;
    STAMP_INIT();
    STAMP(0);
    if (tid < 2) sm.g.bad[tid] = 0u;
    if (tid < 128) (&sm.g.poll[0][0])[tid] = 0u;
#define WDELAY 8
    for (int i = 0; i < WDELAY; ++i) __builtin_amdgcn_s_sleep(2);
    i32x8 Bf[4][4];
    {
        const uint8_t* wp = p.w + (size_t)(512 * w + 16 * q) * NS + 64 * j + 4 * r;
#pragma unroll
        for (int s = 0; s < 4; ++s) {
            unsigned t[32];
#pragma unroll
            for (int h = 0; h < 2; ++h)
#pragma unroll
                for (int k = 0; k < 16; ++k)
                    t[h * 16 + k] = *(const unsigned*)(wp + (size_t)(128 * s + 64 * h + k) * NS);
#pragma unroll
            for (int jj = 0; jj < 8; ++jj) {
                unsigned o0, o1, o2, o3;
                transpose4(t[4 * jj], t[4 * jj + 1], t[4 * jj + 2], t[4 * jj + 3], o0, o1, o2, o3);
                Bf[s][0][jj] = o0; Bf[s][1][jj] = o1; Bf[s][2][jj] = o2; Bf[s][3][jj] = o3;
            }
        }
    }
    const float scale = p.sx[0] * p.sw[0];
    const unsigned call = draw_call(p);
    __syncthreads();
    STAMP(1);
    unsigned* tmo = (unsigned*)(p.me + TMO_OFF);
    const uint8_t* stage = p.me + STAGE_OFF + (size_t)(call & 1u) * SLOT_BYTES + w * PIECE_BYTES;
    float* outp = p.out + (size_t)(tid >> 5) * NS + 64 * j + (tid & 31) * 2;
    const int nloc = g >= 16 ? 2 : 1;
    const int nrem = g < 8 ? 10 : 9;
    bool bad = false;

    auto finish = [&](const f32x4 (&acc)[4], int buf, int row0) {
#pragma unroll
        for (int t = 0; t < 4; ++t) {
            f32x4 v = f32x4{acc[0][t], acc[1][t], acc[2][t], acc[3][t]};
            asm volatile("" : "+v"(v));
            sm.g.red[buf][w][4 * q + t][r] = v;
        }
        if (bad) sm.g.bad[buf] = 1u;
        __syncthreads();
        const bool anybad = sm.g.bad[buf] != 0u;
        const f32x2* rp = (const f32x2*)&sm.g.red[buf][0][0][0] + tid;
        f32x2 v = rp[0];
#pragma unroll
        for (int k = 1; k < 8; ++k) v += rp[k * 512];
        v *= scale;
        v[0] = fmaxf(v[0], 0.f);
        v[1] = fmaxf(v[1], 0.f);
        if (anybad) v = f32x2{__builtin_nanf(""), __builtin_nanf("")};
        asm volatile("global_store_dwordx2 %0, %1, off\n\ts_nop 0" :: "v"(outp + (size_t)row0 * NS), "v"(v) : "memory");
    };

    auto item = [&](int k, const uint8_t*& pc, const unsigned*& flag, unsigned& seq, int& row0) {
        const int n = g + NGROUP * k;
        const int i = n % 7, b = n / 7;
        const unsigned srcr = (unsigned)(p.rank + 1 + i) & 7u;
        pc = uniform_ptr(stage + (size_t)srcr * SSHARD_BYTES + (size_t)b * SBLK_BYTES);
        flag = (const unsigned*)uniform_ptr(p.me + FLAG_OFF + (size_t)(srcr * 8 + w) * 128);
        seq = (call - 1u) * NBLK + b + 1u;
        row0 = (int)srcr * MS + 16 * b;
    };
    const unsigned poll_lds = (unsigned)__builtin_amdgcn_readfirstlane((int)(unsigned)(uintptr_t)&sm.g.poll[w][0]);
    auto poll = [&](int k, const unsigned* flag) {
        unsigned keep; unsigned long long sv;
        asm volatile(
            "s_mov_b32 %0, m0\n\ts_mov_b64 %1, exec\n\ts_mov_b32 m0, %4\n\ts_mov_b64 exec, 1\n\t"
            "global_load_lds_dword %2, %3 sc0 sc1\n\t"
            "s_mov_b64 exec, %1\n\ts_mov_b32 m0, %0"
            : "=&s"(keep), "=&s"(sv) : "v"(0u), "s"(flag), "s"(poll_lds + 4u * (unsigned)k) : "memory");
    };
    auto polled = [&](int k) -> unsigned {
        return (unsigned)__builtin_amdgcn_readfirstlane((int)__hip_atomic_load(&sm.g.poll[w][k], __ATOMIC_RELAXED, __HIP_MEMORY_SCOPE_WORKGROUP));
    };
    const unsigned* ab = (const unsigned*)&sm.g.abuf[w][0];
    const unsigned ab_lds = (unsigned)__builtin_amdgcn_readfirstlane((int)(unsigned)(uintptr_t)ab);
    auto fetch_piece = [&](const uint8_t* pc) {
        unsigned keep; unsigned long long sv;
        asm volatile(
            "s_mov_b32 %0, m0\n\t"
            "s_mov_b32 m0, %5\n\ts_nop 0\n\t"
            "global_load_lds_dwordx4 %2, %3 sc0 sc1\n\t"
            "global_load_lds_dwordx4 %2, %3 offset:1024 sc0 sc1\n\t"
            "global_load_lds_dwordx4 %2, %3 offset:2048 sc0 sc1\n\t"
            "global_load_lds_dwordx4 %2, %3 offset:3072 sc0 sc1\n\t"
            "s_add_u32 m0, %5, 4096\n\ts_nop 0\n\t"
            "global_load_lds_dwordx4 %2, %4 sc0 sc1\n\t"
            "global_load_lds_dwordx4 %2, %4 offset:1024 sc0 sc1\n\t"
            "global_load_lds_dwordx4 %2, %4 offset:2048 sc0 sc1\n\t"
            "s_mov_b64 %1, exec\n\ts_mov_b64 exec, 0xffff\n\t"
            "global_load_lds_dwordx4 %2, %4 offset:3072 sc0 sc1\n\t"
            "s_mov_b64 exec, %1\n\t"
            "s_mov_b32 m0, %0"
            : "=&s"(keep), "=&s"(sv) : "v"(l * 16), "s"(pc), "s"(pc + 4096), "s"(ab_lds) : "memory", "scc");
    };
    {
        const uint8_t* pc; const unsigned* flag; unsigned seq; int row0;
        item(0, pc, flag, seq, row0); poll(0, flag);
        item(1, pc, flag, seq, row0); poll(1, flag);
    }

    for (int it = 0; it < nloc; ++it) {
        const int lb = g + 8 * it;
        const uint8_t* src = p.x + (size_t)(16 * lb + r) * KD + 512 * w + 16 * q;
        STAMP(2 + 2 * it);
        u32x4 Af[8];
        asm volatile(
            "global_load_dwordx4 %0, %8, off\n\t"
            "global_load_dwordx4 %1, %8, off offset:64\n\t"
            "global_load_dwordx4 %2, %8, off offset:128\n\t"
            "global_load_dwordx4 %3, %8, off offset:192\n\t"
            "global_load_dwordx4 %4, %8, off offset:256\n\t"
            "global_load_dwordx4 %5, %8, off offset:320\n\t"
            "global_load_dwordx4 %6, %8, off offset:384\n\t"
            "global_load_dwordx4 %7, %8, off offset:448\n\t"
            "s_waitcnt vmcnt(0)"
            : "=&v"(Af[0]), "=&v"(Af[1]), "=&v"(Af[2]), "=&v"(Af[3]), "=&v"(Af[4]), "=&v"(Af[5]), "=&v"(Af[6]), "=&v"(Af[7])
            : "v"(src) : "memory");
        f32x4 acc[4];
#pragma unroll
        for (int i = 0; i < 4; ++i) acc[i] = f32x4{0.f, 0.f, 0.f, 0.f};
#pragma unroll
        for (int s = 0; s < 4; ++s) {
            const i32x8 a = join8(Af[2 * s], Af[2 * s + 1]);
#pragma unroll
            for (int i = 0; i < 4; ++i)
                acc[i] = __builtin_amdgcn_mfma_scale_f32_16x16x128_f8f6f4(a, Bf[s][i], acc[i], 0, 0, 0, 0, 0, 0);
        }
        finish(acc, it & 1, p.rank * MS + 16 * lb);
        STAMP(3 + 2 * it);
    }

    unsigned* mk = sm.g.mask[w];
    bool issued = false;
#pragma unroll 1
    for (int k = 0; k < nrem; ++k) {
        const uint8_t* pc; const unsigned* flag; unsigned seq; int row0;
        item(k, pc, flag, seq, row0);
        if (issued) {
            WAITV(2);
        } else {
            if (!bad && polled(k) < seq) {
                const unsigned long long t0 = __builtin_amdgcn_s_memrealtime();
                unsigned itn = 0;
                for (;;) {
                    poll(k, flag);
                    __builtin_amdgcn_s_sleep(3);
                    if (polled(k) >= seq) break;
                    if ((++itn & 15u) == 0u &&
                        (ld_agent(tmo) == call || __builtin_amdgcn_s_memrealtime() - t0 > DEADLINE_TICKS)) {
                        if (l == 0) __hip_atomic_store(tmo, call, __ATOMIC_RELAXED, __HIP_MEMORY_SCOPE_AGENT);
                        bad = true;
                        break;
                    }
                }
            }
            STAMP(2 + 2 * (k + nloc));
            WSTAMP(0);
            fetch_piece(pc);
            WAITV(0);
        }
        DSTAMP(0);
        WSTAMP(1);
        const uint8_t* pcn; const unsigned* flagn; unsigned seqn; int row0n;
        item(k + 1 < nrem ? k + 1 : k, pcn, flagn, seqn, row0n);
        poll(k + 1 < nrem ? k + 1 : k, flagn);
        *(u32x4*)&mk[l * 4] = u32x4{0u, 0u, 0u, 0u};
        unsigned ex = ab[LIST_OFF / 4 + l];
        u32x4 lo[4];
        u32x3 hi[4];
#pragma unroll
        for (int s = 0; s < 4; ++s) {
            lo[s] = *(const u32x4*)(ab + s * 256 + l * 4);
            const unsigned* h = ab + HI_OFF / 4 + s * 192 + l * 3;
            hi[s] = u32x3{h[0], h[1], h[2]};
        }
        asm volatile("s_waitcnt lgkmcnt(0)" : "+v"(ex), "+v"(lo[0]), "+v"(lo[1]), "+v"(lo[2]), "+v"(lo[3]),
                     "+v"(hi[0]), "+v"(hi[1]), "+v"(hi[2]), "+v"(hi[3]) :: "memory");
        DSTAMP(1);
        issued = k + 1 < nrem && !bad && polled(k + 1) >= seqn;
        if (issued) fetch_piece(pcn);
        DSTAMP(2);
        const unsigned h0 = (unsigned)__builtin_amdgcn_readlane((int)ex, 0), h1 = (unsigned)__builtin_amdgcn_readlane((int)ex, 1);
        const unsigned nv = h0 & 0xFFFFu, rawb = h0 >> 16;
        const unsigned i0 = 2u * (unsigned)l - 4u;
        if (i0 < nv) atomicOr(&mk[(ex >> 5) & 255u], 1u << (ex & 31u));
        if (i0 + 1u < nv && l >= 2) atomicOr(&mk[(ex >> 21) & 255u], 1u << ((ex >> 16) & 31u));
        unsigned mv[4];
#pragma unroll
        for (int s = 0; s < 4; ++s) mv[s] = mk[s * 64 + l];
        DSTAMP(3);
        f32x4 acc[4];
#pragma unroll
        for (int i2 = 0; i2 < 4; ++i2) acc[i2] = f32x4{0.f, 0.f, 0.f, 0.f};
        if (rawb == 0u) {
            i32x8 a = unpack_unit(lo[0], hi[0], mv[0], (h1 & 0xFFu) * 0x01010101u);
#define SB() __builtin_amdgcn_sched_barrier(0)
#define UF(j) an[j] = (int)(((d[j] & 0xBFBFBFBFu) + A) ^ ((m << (6 - j)) & 0x40404040u))
#define UG(j) gg = and_or(d[j] >> (6 - j), 0x01010101u << j, gg)
#define MM(s, i) acc[i] = __builtin_amdgcn_mfma_scale_f32_16x16x128_f8f6f4(a, Bf[s][i], acc[i], 0, 0, 0, 0, 0, 0)
#pragma unroll
            for (int s = 0; s < 3; ++s) {
                const unsigned d[7] = {lo[s + 1][0], lo[s + 1][1], lo[s + 1][2], lo[s + 1][3], hi[s + 1][0], hi[s + 1][1], hi[s + 1][2]};
                const unsigned m = mv[s + 1], A = ((h1 >> (8 * (s + 1))) & 0xFFu) * 0x01010101u;
                i32x8 an;
                unsigned gg;
                SB(); MM(s, 0); SB();
                UF(0); UF(1); gg = (d[6] << 1) & 0x80808080u; UG(0);
                SB(); MM(s, 1); SB();
                UF(2); UF(3); UG(1); UG(2);
                SB(); MM(s, 2); SB();
                UF(4); UF(5); UG(3); UG(4);
                SB(); MM(s, 3); SB();
                UF(6); UG(5);
                an[7] = (int)((gg + A) ^ ((m >> 1) & 0x40404040u));
                a = an;
            }
            SB(); MM(3, 0); MM(3, 1); MM(3, 2); MM(3, 3); SB();
#undef SB
#undef UF
#undef UG
#undef MM
        } else {
#pragma unroll
            for (int s = 0; s < 4; ++s) {
                i32x8 a;
                if ((rawb >> s) & 1u) {
                    unsigned d7;
                    asm volatile("global_load_dword %0, %1, %2 sc0 sc1\n\ts_waitcnt vmcnt(0)"
                                 : "=&v"(d7) : "v"(RAW7_OFF + s * 256 + l * 4), "s"(pc) : "memory");
                    a[0] = lo[s][0]; a[1] = lo[s][1]; a[2] = lo[s][2]; a[3] = lo[s][3];
                    a[4] = hi[s][0]; a[5] = hi[s][1]; a[6] = hi[s][2]; a[7] = d7;
                } else {
                    a = unpack_unit(lo[s], hi[s], mv[s], ((h1 >> (8 * s)) & 0xFFu) * 0x01010101u);
                }
#pragma unroll
                for (int i2 = 0; i2 < 4; ++i2)
                    acc[i2] = __builtin_amdgcn_mfma_scale_f32_16x16x128_f8f6f4(a, Bf[s][i2], acc[i2], 0, 0, 0, 0, 0, 0);
            }
        }
        DSTAMP(4);
        WSTAMP(2);
        if (!issued && k + 1 < nrem && !bad && polled(k + 1) >= seqn) { fetch_piece(pcn); issued = true; }
        {
            const int k2 = k + 2 < nrem ? k + 2 : k;
            const uint8_t* pc2; const unsigned* flag2; unsigned seq2; int row2;
            item(k2, pc2, flag2, seq2, row2);
            poll(k2, flag2);
        }
        finish(acc, (k + nloc) & 1, row0);
        DSTAMP(5);
        STAMP(3 + 2 * (k + nloc));
    }
    STAMP(39);
    STAMP_FLUSH();
}

__global__ void __launch_bounds__(NTHREADS, 2) ag_gemm_kernel(Params p) {
    __shared__ Smem sm;
    const int xcd = blockIdx.x & 7, slot = blockIdx.x >> 3;
    if (slot < 7) pusher(p, sm, slot, xcd);
    else if (slot >= 8) gemm_wg(p, sm, 3 * xcd + ((slot - 8) >> 3), (slot - 8) & 7);
}

}

extern "C" void kernel_launch(int rank, int nranks, ncclComm_t comm, hipStream_t stream,
                              void* const* d_in, const int* in_sizes, int n_in,
                              void* d_out, int out_size, const int* in_dtypes, int out_dtype,
                              void* const* d_out_all, void* const* d_ws_all, size_t ws_bytes) {
    (void)comm; (void)in_dtypes; (void)out_dtype; (void)d_out_all;
    if (nranks != NR || n_in != 4 || (unsigned)rank >= (unsigned)NR || ws_bytes < WS_NEED ||
        in_sizes[0] != MS * KD || in_sizes[1] != KD * NS || out_size != NR * MS * NS) {
        fprintf(stderr, "kernel_launch: unexpected problem shape\n");
        return;
    }
    Params p;
    p.x = (const uint8_t*)d_in[0];
    p.w = (const uint8_t*)d_in[1];
    p.sx = (const float*)d_in[2];
    p.sw = (const float*)d_in[3];
    p.out = (float*)d_out;
    p.me = (uint8_t*)d_ws_all[rank];
    for (int i = 0; i < 7; ++i) p.peer[i] = (uint8_t*)d_ws_all[(rank + 1 + i) % NR];
    p.rank = rank;
    p.pad = 0;
    hipLaunchKernelGGL(ag_gemm_kernel, dim3(NGRID), dim3(NTHREADS), 0, stream, p);
    hipError_t e = hipGetLastError();
    if (e != hipSuccess) fprintf(stderr, "launch error %s\n", hipGetErrorString(e));
}
```

```cpp
#include <hip/hip_runtime.h>
#include <rccl/rccl.h>
#include <cstdio>
#include <cstdint>

typedef unsigned u32x4 __attribute__((ext_vector_type(4)));
typedef unsigned u32x3 __attribute__((ext_vector_type(3)));
typedef int i32x8 __attribute__((ext_vector_type(8)));
typedef float f32x4 __attribute__((ext_vector_type(4)));
typedef float f32x2 __attribute__((ext_vector_type(2)));

namespace {

constexpr int NR = 8;
constexpr int KD = 4096;
constexpr int MS = 512;
constexpr int NS = 512;
constexpr int NBLK = 32;
constexpr int BLK_BYTES = 16 * KD;
constexpr int NGRID = 256;
constexpr int NTHREADS = 512;
constexpr int NGROUP = 24;

constexpr int HI_OFF = 4096, LIST_OFF = 7168, RAW7_OFF = 7424;
constexpr int PIECE_BYTES = 8448;
constexpr int SBLK_BYTES = 8 * PIECE_BYTES;
constexpr int SSHARD_BYTES = NBLK * SBLK_BYTES;
constexpr unsigned LIST_CAP = 124;
constexpr int ABUF_BYTES = 8192;
#define PLAIN_HEAD 1
#define PLAIN_TAIL 2
__host__ __device__ constexpr bool plain_block(int b) { return b < PLAIN_HEAD || b >= NBLK - PLAIN_TAIL; }
constexpr int MAXITEMS = 12;

constexpr size_t CNT_OFF = 0;
constexpr size_t TMO_OFF = 32768;
constexpr size_t FLAG_OFF = 65536;
constexpr size_t DBG_OFF = 131072;
constexpr size_t STAGE_OFF = 1u << 20;
constexpr size_t SLOT_BYTES = (size_t)NR * SSHARD_BYTES;
constexpr size_t WS_NEED = STAGE_OFF + 2 * SLOT_BYTES;

constexpr unsigned long long DEADLINE_TICKS = 40000000ull;

struct Params {
    const uint8_t* x;
    const uint8_t* w;
    const float* sx;
    const float* sw;
    float* out;
    uint8_t* me;
    uint8_t* peer[7];
    int rank;
    int pad;
};

__device__ __forceinline__ unsigned ld_sys(const unsigned* p) {
    return __hip_atomic_load(p, __ATOMIC_RELAXED, __HIP_MEMORY_SCOPE_SYSTEM);
}
__device__ __forceinline__ unsigned ld_agent(const unsigned* p) {
    return __hip_atomic_load(p, __ATOMIC_RELAXED, __HIP_MEMORY_SCOPE_AGENT);
}

#define STAMP(idx) do {} while (0)
#define STAMP_INIT() do {} while (0)
#define STAMP_FLUSH() do {} while (0)

#define DSTAMP(idx) do {} while (0)
#define WSTAMP(idx) do {} while (0)
#define WAITV(n) asm volatile("s_waitcnt vmcnt(" #n ")" ::: "memory")

__device__ __forceinline__ unsigned draw_call(const Params& p) {
    unsigned* cnt = (unsigned*)(p.me + CNT_OFF + (size_t)blockIdx.x * 128) + (threadIdx.x >> 6);
    unsigned c = 0;
    if ((threadIdx.x & 63) == 0) c = __hip_atomic_fetch_add(cnt, 1u, __ATOMIC_RELAXED, __HIP_MEMORY_SCOPE_AGENT) + 1u;
    return (unsigned)__builtin_amdgcn_readfirstlane((int)c);
}

__device__ __forceinline__ const uint8_t* uniform_ptr(const uint8_t* q) {
    const unsigned long long v = (unsigned long long)q;
    const unsigned lo = (unsigned)__builtin_amdgcn_readfirstlane((int)(unsigned)v);
    const unsigned hi = (unsigned)__builtin_amdgcn_readfirstlane((int)(unsigned)(v >> 32));
    return (const uint8_t*)(((unsigned long long)hi << 32) | lo);
}

struct alignas(16) Smem {
    union {
        struct {
            f32x4 red[2][8][16][16];
            u32x4 abuf[8][ABUF_BYTES / 16];
            unsigned mask[8][256];
            unsigned poll[8][16];
            unsigned bad[2];
        } g;
        struct {
            u32x4 ring[4][4][128];
            unsigned short sub[NBLK][4][32];
            unsigned short meta[NBLK][4];
            unsigned packed[NBLK];
            unsigned done[NBLK];
        } s;
    };
};

__device__ __forceinline__ unsigned and_or(unsigned a, unsigned m, unsigned c) { return (a & m) | c; }
__device__ __forceinline__ unsigned bfi(unsigned m, unsigned a, unsigned b) { return (a & m) | (b & ~m); }

__device__ __forceinline__ unsigned wave_max(unsigned v) {
    v = max(v, (unsigned)__builtin_amdgcn_update_dpp(0, (int)v, 0x111, 0xf, 0xf, true));
    v = max(v, (unsigned)__builtin_amdgcn_update_dpp(0, (int)v, 0x112, 0xf, 0xf, true));
    v = max(v, (unsigned)__builtin_amdgcn_update_dpp(0, (int)v, 0x114, 0xf, 0xf, true));
    v = max(v, (unsigned)__builtin_amdgcn_update_dpp(0, (int)v, 0x118, 0xf, 0xf, true));
    v = max(v, (unsigned)__builtin_amdgcn_update_dpp(0, (int)v, 0x142, 0xa, 0xf, true));
    v = max(v, (unsigned)__builtin_amdgcn_update_dpp(0, (int)v, 0x143, 0xc, 0xf, true));
    return (unsigned)__builtin_amdgcn_readlane((int)v, 63);
}
__device__ __forceinline__ unsigned wave_sum(unsigned v) {
    v += (unsigned)__builtin_amdgcn_update_dpp(0, (int)v, 0x111, 0xf, 0xf, true);
    v += (unsigned)__builtin_amdgcn_update_dpp(0, (int)v, 0x112, 0xf, 0xf, true);
    v += (unsigned)__builtin_amdgcn_update_dpp(0, (int)v, 0x114, 0xf, 0xf, true);
    v += (unsigned)__builtin_amdgcn_update_dpp(0, (int)v, 0x118, 0xf, 0xf, true);
    v += (unsigned)__builtin_amdgcn_update_dpp(0, (int)v, 0x142, 0xa, 0xf, true);
    v += (unsigned)__builtin_amdgcn_update_dpp(0, (int)v, 0x143, 0xc, 0xf, true);
    return (unsigned)__builtin_amdgcn_readlane((int)v, 63);
}

__device__ __forceinline__ void fetch_unit(const uint8_t* src, unsigned lds) {
    unsigned keep;
    asm volatile(
        "s_mov_b32 %0, m0\n\t"
        "s_mov_b32 m0, %3\n\t"
        "s_nop 0\n\t"
        "global_load_lds_dwordx4 %1, off\n\t"
        "s_add_u32 m0, %3, 1024\n\t"
        "s_nop 0\n\t"
        "global_load_lds_dwordx4 %2, off\n\t"
        "s_mov_b32 m0, %0"
        : "=&s"(keep) : "v"(src), "v"(src + 64), "s"(lds) : "memory", "scc");
}

struct Packed {
    u32x4 lo;
    u32x3 hi;
    unsigned d7;
    bool raw;
};

constexpr unsigned SUB_CAP = 31;
__device__ __forceinline__ void pack_unit(const u32x4* slot, Smem& sm, int t, int s, int l, unsigned& a, bool& refresh, Packed& o) {
    const u32x4 R0 = slot[l], R1 = slot[64 + l];
    const unsigned f[8] = {R0[0], R0[1], R0[2], R0[3], R1[0], R1[1], R1[2], R1[3]};
    if (refresh) {
        const unsigned e0 = max(f[0] & 0x7Fu, f[4] & 0x7Fu), e1 = max((f[0] >> 8) & 0x7Fu, (f[4] >> 8) & 0x7Fu);
        const unsigned e2 = max((f[0] >> 16) & 0x7Fu, (f[4] >> 16) & 0x7Fu), e3 = max((f[0] >> 24) & 0x7Fu, (f[4] >> 24) & 0x7Fu);
        const unsigned mx = wave_max(max(max(e0, e1), max(e2, e3)));
        a = mx > 63u ? mx - 63u : 0u;
    }
    const unsigned na = (128u - a) * 0x01010101u;
    unsigned tt[8], m = 0;
#pragma unroll
    for (int j = 0; j < 8; ++j) tt[j] = (f[j] & 0x7F7F7F7Fu) + na;
#pragma unroll
    for (int j = 0; j < 7; ++j) m = and_or(tt[j] >> (6 - j), 0x01010101u << j, m);
    m = and_or(tt[7] << 1, 0x80808080u, m);
    unsigned short* l16 = sm.s.sub[t][s];
    const unsigned tag = (unsigned)((s << 11) | (l << 5));
    unsigned n = 0, mm = m;
    for (;;) {
        const unsigned long long bal = __ballot(mm != 0u);
        if (bal == 0ull || n > SUB_CAP) break;
        const unsigned idx = n + __builtin_amdgcn_mbcnt_hi((unsigned)(bal >> 32), __builtin_amdgcn_mbcnt_lo((unsigned)bal, 0u));
        if (mm != 0u) l16[min(idx, SUB_CAP)] = (unsigned short)(tag | (unsigned)(__ffs(mm) - 1));
        mm &= mm - 1u;
        n += (unsigned)__popcll(bal);
    }
    o.raw = n > SUB_CAP;
    refresh = n > 27u;
    {
        unsigned long long sv;
        const unsigned meta_lds = (unsigned)(uintptr_t)&sm.s.meta[t][s], cnt_lds = (unsigned)(uintptr_t)&sm.s.packed[t];
        asm volatile("s_mov_b64 %0, exec\n\ts_mov_b64 exec, 1\n\tds_write_b16 %1, %2\n\tds_add_u32 %3, %4\n\ts_mov_b64 exec, %0"
                     : "=&s"(sv) : "v"(meta_lds), "v"((a << 8) | (o.raw ? 0xFFu : n)), "v"(cnt_lds), "v"(1u) : "memory");
    }
    const unsigned p7 = and_or(tt[7], 0x3F3F3F3Fu, f[7] & 0x80808080u);
    unsigned d[7];
#pragma unroll
    for (int j = 0; j < 7; ++j) {
        const unsigned sp = j < 6 ? (p7 << (6 - j)) : (p7 >> 1);
        d[j] = and_or(sp, 0x40404040u, and_or(tt[j], 0x3F3F3F3Fu, f[j] & 0x80808080u));
    }
    if (o.raw) {
        o.lo = R0;
        o.hi = u32x3{f[4], f[5], f[6]};
    } else {
        o.lo = u32x4{d[0], d[1], d[2], d[3]};
        o.hi = u32x3{d[4], d[5], d[6]};
    }
    o.d7 = f[7];
}

__device__ __forceinline__ void store_unit(const Packed& o, uint8_t* piece, int s, int l) {
    asm volatile(
        "global_store_dwordx4 %0, %1, %4 sc0 sc1\n\t"
        "global_store_dwordx3 %2, %3, %4 sc0 sc1\n\t"
        "s_nop 1"
        :: "v"(s * 1024 + l * 16), "v"(o.lo), "v"(HI_OFF + s * 768 + l * 12), "v"(o.hi), "s"(piece) : "memory");
    if (o.raw)
        asm volatile("global_store_dword %0, %1, %2 sc0 sc1\n\ts_nop 1"
                     :: "v"(RAW7_OFF + s * 256 + l * 4), "v"(o.d7), "s"(piece) : "memory");
}

__device__ __forceinline__ bool lds_wait(const unsigned* word, unsigned value, unsigned long long t0) {
    unsigned itn = 0;
    while (__hip_atomic_load(word, __ATOMIC_RELAXED, __HIP_MEMORY_SCOPE_WORKGROUP) < value) {
        if ((++itn & 1023u) == 0u && __builtin_amdgcn_s_memrealtime() - t0 > DEADLINE_TICKS) return false;
        __builtin_amdgcn_s_sleep(1);
    }
    asm volatile("" ::: "memory");
    return true;
}

__device__ __forceinline__ void pusher(const Params& p, Smem& sm, int dd, int w) {
    const int tid = threadIdx.x;
    const int s = __builtin_amdgcn_readfirstlane(tid >> 6);
    const int l = tid & 63, r = l & 15, q = l >> 4;
#define WIN 2
    STAMP_INIT();
    STAMP(0);
    const uint8_t* src = p.x + (size_t)r * KD + 512 * w + 128 * (s & 3) + 16 * q;
    u32x4* ring = &sm.s.ring[s & 3][0][0];
    const unsigned ring_lds = (unsigned)__builtin_amdgcn_readfirstlane((int)(unsigned)(uintptr_t)ring);
    if (s < 4) {
        fetch_unit(src, ring_lds);
        fetch_unit(src + BLK_BYTES, ring_lds + 2048u);
        if (WIN == 3) fetch_unit(src + 2 * BLK_BYTES, ring_lds + 4096u);
    }
    if (tid < NBLK) { sm.s.packed[tid] = 0u; sm.s.done[tid] = 0u; }
    asm volatile("s_waitcnt lgkmcnt(0)" ::: "memory");
    __builtin_amdgcn_s_barrier();
    asm volatile("" ::: "memory");
    if (s > 4) return;
    uint8_t* pw = p.peer[dd];
    if (s == 4) {
        unsigned* flag = (unsigned*)(pw + FLAG_OFF + (size_t)(p.rank * 8 + w) * 128);
        const unsigned call = draw_call(p);
        const unsigned base = (call - 1u) * NBLK;
        uint8_t* dst = (uint8_t*)uniform_ptr(pw + STAGE_OFF + (size_t)(call & 1u) * SLOT_BYTES + (size_t)p.rank * SSHARD_BYTES +
                                             w * PIECE_BYTES + LIST_OFF);
        const unsigned long long t0 = __builtin_amdgcn_s_memrealtime();
#define PUBLISH(t) do { unsigned long long sv_; \
            asm volatile("s_mov_b64 %0, exec\n\ts_mov_b64 exec, 1\n\tglobal_atomic_umax %1, %2, off sc1\n\ts_mov_b64 exec, %0" \
                         : "=&s"(sv_) : "v"(flag), "v"(base + (unsigned)(t) + 1u) : "memory"); } while (0)
#pragma unroll 1
        for (int t = 0; t < NBLK; ++t) {
            if (!lds_wait(&sm.s.packed[t], 4u, t0)) return;
            const unsigned m01 = ((const unsigned*)sm.s.meta[t])[0], m23 = ((const unsigned*)sm.s.meta[t])[1];
            const unsigned c0 = m01 & 0xFFu, c1 = (m01 >> 16) & 0xFFu, c2 = m23 & 0xFFu, c3 = (m23 >> 16) & 0xFFu;
            const unsigned rawb = (c0 == 0xFFu ? 1u : 0u) | (c1 == 0xFFu ? 2u : 0u) | (c2 == 0xFFu ? 4u : 0u) | (c3 == 0xFFu ? 8u : 0u);
            const unsigned p1 = c0 == 0xFFu ? 0u : c0, p2 = p1 + (c1 == 0xFFu ? 0u : c1), p3 = p2 + (c2 == 0xFFu ? 0u : c2);
            const unsigned nv = p3 + (c3 == 0xFFu ? 0u : c3);
            unsigned v = 0;
#pragma unroll
            for (int h = 0; h < 2; ++h) {
                const unsigned i = 2u * (unsigned)l - 4u + (unsigned)h;
                const unsigned u = (i >= p1 ? 1u : 0u) + (i >= p2 ? 1u : 0u) + (i >= p3 ? 1u : 0u);
                const unsigned jx = i - (u == 0u ? 0u : u == 1u ? p1 : u == 2u ? p2 : p3);
                v |= (unsigned)sm.s.sub[t][u][jx & 31u] << (16 * h);
            }
            if (l == 0) v = nv | (rawb << 16);
            if (l == 1) v = ((m01 >> 8) & 0xFFu) | ((m01 >> 24) << 8) | (((m23 >> 8) & 0xFFu) << 16) | ((m23 >> 24) << 24);
            const bool plain = plain_block(t);
            const unsigned nl = plain ? 1u : 2u + (nv + 1u) / 2u;
            const unsigned long long em = nl >= 64u ? ~0ull : ((1ull << nl) - 1ull);
            unsigned long long sv;
            asm volatile("s_mov_b64 %0, exec\n\ts_mov_b64 exec, %4\n\tglobal_store_dword %1, %2, %3 sc0 sc1\n\ts_mov_b64 exec, %0"
                         : "=&s"(sv) : "v"(l * 4 + (plain ? PIECE_BYTES - LIST_OFF - 4 : 0)), "v"(v), "s"(dst + (size_t)t * SBLK_BYTES), "s"(em) : "memory");
            if (t >= 2) {
                if (!lds_wait(&sm.s.done[t - 2], 4u, t0)) return;
                if (t >= 4) WAITV(4); else WAITV(2);
                PUBLISH(t - 2);
            }
        }
        if (!lds_wait(&sm.s.done[NBLK - 2], 4u, t0)) return;
        WAITV(3);
        PUBLISH(NBLK - 2);
        if (!lds_wait(&sm.s.done[NBLK - 1], 4u, t0)) return;
        WAITV(2);
        PUBLISH(NBLK - 1);
#undef PUBLISH
        return;
    }
#define PACE 1648
    const unsigned call = draw_call(p);
    uint8_t* dst = (uint8_t*)uniform_ptr(pw + STAGE_OFF + (size_t)(call & 1u) * SLOT_BYTES + (size_t)p.rank * SSHARD_BYTES +
                                         w * PIECE_BYTES);
    WAITV(0);
    STAMP(1);
    const unsigned long long tstart = __builtin_amdgcn_s_memrealtime();
    unsigned win_a = 0;
    bool win_refresh = true;
#define LDS_COUNT(word) do { unsigned long long sv_; \
        asm volatile("s_mov_b64 %0, exec\n\ts_mov_b64 exec, 1\n\tds_add_u32 %1, %2\n\ts_mov_b64 exec, %0" \
                     : "=&s"(sv_) : "v"((unsigned)(uintptr_t)&(word)), "v"(1u) : "memory"); } while (0)
#define SIGNAL(t) LDS_COUNT(sm.s.done[t])
#pragma unroll 1
    for (int t = 0; t < NBLK; ++t) {
        if (WIN == 2) { if (t >= 2) WAITV(6); }
        else { if (t >= 3) WAITV(10); }
        Packed o;
        const bool plain = plain_block(t);
        u32x4 r0, r1;
        if (plain) {
            r0 = ring[(t & 3) * 128 + l];
            r1 = ring[(t & 3) * 128 + 64 + l];
            asm volatile("s_waitcnt lgkmcnt(0)" : "+v"(r0), "+v"(r1) :: "memory");
            LDS_COUNT(sm.s.packed[t]);
        } else {
            pack_unit(ring + (t & 3) * 128, sm, t, s, l, win_a, win_refresh, o);
        }
        const int tn = t + WIN < NBLK ? t + WIN : t;
        fetch_unit(src + (size_t)tn * BLK_BYTES, ring_lds + (unsigned)((t + WIN) & 3) * 2048u);
        if (WIN == 2) { if (t >= 2) { WAITV(6); SIGNAL(t - 2); if (t == 2) STAMP(4); if (t == 3) STAMP(5); if (t == 6) STAMP(6); } }
        else { if (t >= 3) { WAITV(10); SIGNAL(t - 3); } }
        if (PACE > 0) {
            const int nplain = (t < PLAIN_HEAD ? t : PLAIN_HEAD) + (t > NBLK - PLAIN_TAIL ? t - (NBLK - PLAIN_TAIL) : 0);
            const unsigned long long due = tstart + (((unsigned long long)(8 * t + nplain) * PACE) >> 7);
            while (__builtin_amdgcn_s_memrealtime() < due) __builtin_amdgcn_s_sleep(1);
        }
        if (plain)
            asm volatile(
                "global_store_dwordx4 %0, %1, %3 sc0 sc1\n\t"
                "global_store_dwordx4 %0, %2, %3 offset:1024 sc0 sc1\n\t"
                "s_nop 1"
                :: "v"(s * 2048 + l * 16), "v"(r0), "v"(r1), "s"(dst + (size_t)t * SBLK_BYTES) : "memory");
        else
            store_unit(o, dst + (size_t)t * SBLK_BYTES, s, l);
        if (t == 15) STAMP(2);
    }
    if (WIN == 3) { WAITV(8); SIGNAL(NBLK - 3); }
    WAITV(4); SIGNAL(NBLK - 2);
    WAITV(0); SIGNAL(NBLK - 1);
    STAMP(3);
#undef SIGNAL
#undef LDS_COUNT
}

__device__ __forceinline__ void transpose4(unsigned a, unsigned b, unsigned c, unsigned d,
                                           unsigned& o0, unsigned& o1, unsigned& o2, unsigned& o3) {
    const unsigned t0 = __builtin_amdgcn_perm(b, a, 0x05010400u);
    const unsigned t1 = __builtin_amdgcn_perm(b, a, 0x07030602u);
    const unsigned t2 = __builtin_amdgcn_perm(d, c, 0x05010400u);
    const unsigned t3 = __builtin_amdgcn_perm(d, c, 0x07030602u);
    o0 = __builtin_amdgcn_perm(t2, t0, 0x05040100u);
    o1 = __builtin_amdgcn_perm(t2, t0, 0x07060302u);
    o2 = __builtin_amdgcn_perm(t3, t1, 0x05040100u);
    o3 = __builtin_amdgcn_perm(t3, t1, 0x07060302u);
}

__device__ __forceinline__ i32x8 join8(u32x4 lo, u32x4 hi) {
    i32x8 v;
    v[0] = lo[0]; v[1] = lo[1]; v[2] = lo[2]; v[3] = lo[3];
    v[4] = hi[0]; v[5] = hi[1]; v[6] = hi[2]; v[7] = hi[3];
    return v;
}

__device__ __forceinline__ i32x8 unpack_unit(u32x4 lo, u32x3 hi, unsigned m, unsigned A) {
    const unsigned d[7] = {lo[0], lo[1], lo[2], lo[3], hi[0], hi[1], hi[2]};
    i32x8 f;
    unsigned g = (d[6] << 1) & 0x80808080u;
#pragma unroll
    for (int j = 0; j < 6; ++j) g = and_or(d[j] >> (6 - j), 0x01010101u << j, g);
#pragma unroll
    for (int j = 0; j < 7; ++j) f[j] = (int)(((d[j] & 0xBFBFBFBFu) + A) ^ ((m << (6 - j)) & 0x40404040u));
    f[7] = (int)((g + A) ^ ((m >> 1) & 0x40404040u));
    return f;
}

__device__ __forceinline__ void gemm_wg(const Params& p, Smem& sm, int g, int j) {
    const int tid = threadIdx.x, w = __builtin_amdgcn_readfirstlane(tid >> 6), l = tid & 63, r = l & 15, q = l >> 4;
    STAMP_INIT();
    STAMP(0);
    if (tid < 2) sm.g.bad[tid] = 0u;
    if (tid < 128) (&sm.g.poll[0][0])[tid] = 0u;
#define WDELAY 8
    for (int i = 0; i < WDELAY; ++i) __builtin_amdgcn_s_sleep(2);
    i32x8 Bf[4][4];
    {
        const uint8_t* wp = p.w + (size_t)(512 * w + 16 * q) * NS + 64 * j + 4 * r;
#pragma unroll
        for (int s = 0; s < 4; ++s) {
            unsigned t[32];
#pragma unroll
            for (int h = 0; h < 2; ++h)
#pragma unroll
                for (int k = 0; k < 16; ++k)
                    t[h * 16 + k] = *(const unsigned*)(wp + (size_t)(128 * s + 64 * h + k) * NS);
#pragma unroll
            for (int jj = 0; jj < 8; ++jj) {
                unsigned o0, o1, o2, o3;
                transpose4(t[4 * jj], t[4 * jj + 1], t[4 * jj + 2], t[4 * jj + 3], o0, o1, o2, o3);
                Bf[s][0][jj] = o0; Bf[s][1][jj] = o1; Bf[s][2][jj] = o2; Bf[s][3][jj] = o3;
            }
        }
    }
    const float scale = p.sx[0] * p.sw[0];
    const unsigned call = draw_call(p);
    __syncthreads();
    STAMP(1);
    unsigned* tmo = (unsigned*)(p.me + TMO_OFF);
    const uint8_t* stage = p.me + STAGE_OFF + (size_t)(call & 1u) * SLOT_BYTES + w * PIECE_BYTES;
    float* outp = p.out + (size_t)(tid >> 5) * NS + 64 * j + (tid & 31) * 2;
    const int nloc = g >= 16 ? 2 : 1;
    const int nrem = g < 8 ? 10 : 9;
    bool bad = false;

    auto finish = [&](const f32x4 (&acc)[4], int buf, int row0) {
#pragma unroll
        for (int t = 0; t < 4; ++t) {
            f32x4 v = f32x4{acc[0][t], acc[1][t], acc[2][t], acc[3][t]};
            asm volatile("" : "+v"(v));
            sm.g.red[buf][w][4 * q + t][r] = v;
        }
        if (bad) sm.g.bad[buf] = 1u;
        __syncthreads();
        const bool anybad = sm.g.bad[buf] != 0u;
        const f32x2* rp = (const f32x2*)&sm.g.red[buf][0][0][0] + tid;
        f32x2 v = rp[0];
#pragma unroll
        for (int k = 1; k < 8; ++k) v += rp[k * 512];
        v *= scale;
        v[0] = fmaxf(v[0], 0.f);
        v[1] = fmaxf(v[1], 0.f);
        if (anybad) v = f32x2{__builtin_nanf(""), __builtin_nanf("")};
        asm volatile("global_store_dwordx2 %0, %1, off\n\ts_nop 0" :: "v"(outp + (size_t)row0 * NS), "v"(v) : "memory");
    };

    auto item_plain = [&](int k) -> bool { return plain_block((g + NGROUP * k) / 7); };
    auto item = [&](int k, const uint8_t*& pc, const unsigned*& flag, unsigned& seq, int& row0) {
        const int n = g + NGROUP * k;
        const int i = n % 7, b = n / 7;
        const unsigned srcr = (unsigned)(p.rank + 1 + i) & 7u;
        pc = uniform_ptr(stage + (size_t)srcr * SSHARD_BYTES + (size_t)b * SBLK_BYTES);
        flag = (const unsigned*)uniform_ptr(p.me + FLAG_OFF + (size_t)(srcr * 8 + w) * 128);
        seq = (call - 1u) * NBLK + b + 1u;
        row0 = (int)srcr * MS + 16 * b;
    };
    const unsigned poll_lds = (unsigned)__builtin_amdgcn_readfirstlane((int)(unsigned)(uintptr_t)&sm.g.poll[w][0]);
    auto poll = [&](int k, const unsigned* flag) {
        unsigned keep; unsigned long long sv;
        asm volatile(
            "s_mov_b32 %0, m0\n\ts_mov_b64 %1, exec\n\ts_mov_b32 m0, %4\n\ts_mov_b64 exec, 1\n\t"
            "global_load_lds_dword %2, %3 sc0 sc1\n\t"
            "s_mov_b64 exec, %1\n\ts_mov_b32 m0, %0"
            : "=&s"(keep), "=&s"(sv) : "v"(0u), "s"(flag), "s"(poll_lds + 4u * (unsigned)k) : "memory");
    };
    auto polled = [&](int k) -> unsigned {
        return (unsigned)__builtin_amdgcn_readfirstlane((int)__hip_atomic_load(&sm.g.poll[w][k], __ATOMIC_RELAXED, __HIP_MEMORY_SCOPE_WORKGROUP));
    };
    const unsigned* ab = (const unsigned*)&sm.g.abuf[w][0];
    const unsigned ab_lds = (unsigned)__builtin_amdgcn_readfirstlane((int)(unsigned)(uintptr_t)ab);
    auto fetch_piece = [&](const uint8_t* pc, bool plain) {
        unsigned keep; unsigned long long sv;
        asm volatile(
            "s_mov_b32 %0, m0\n\t"
            "s_mov_b32 m0, %5\n\ts_nop 0\n\t"
            "global_load_lds_dwordx4 %2, %3 sc0 sc1\n\t"
            "global_load_lds_dwordx4 %2, %3 offset:1024 sc0 sc1\n\t"
            "global_load_lds_dwordx4 %2, %3 offset:2048 sc0 sc1\n\t"
            "global_load_lds_dwordx4 %2, %3 offset:3072 sc0 sc1\n\t"
            "s_add_u32 m0, %5, 4096\n\ts_nop 0\n\t"
            "global_load_lds_dwordx4 %2, %4 sc0 sc1\n\t"
            "global_load_lds_dwordx4 %2, %4 offset:1024 sc0 sc1\n\t"
            "global_load_lds_dwordx4 %2, %4 offset:2048 sc0 sc1\n\t"
            "s_mov_b64 %1, exec\n\ts_mov_b64 exec, %6\n\t"
            "global_load_lds_dwordx4 %2, %4 offset:3072 sc0 sc1\n\t"
            "s_mov_b64 exec, %1\n\t"
            "s_mov_b32 m0, %0"
            : "=&s"(keep), "=&s"(sv) : "v"(l * 16), "s"(pc), "s"(pc + 4096), "s"(ab_lds), "s"(plain ? ~0ull : 0xffffull) : "memory", "scc");
    };
    {
        const uint8_t* pc; const unsigned* flag; unsigned seq; int row0;
        item(0, pc, flag, seq, row0); poll(0, flag);
        item(1, pc, flag, seq, row0); poll(1, flag);
    }

    for (int it = 0; it < nloc; ++it) {
        const int lb = g + 8 * it;
        const uint8_t* src = p.x + (size_t)(16 * lb + r) * KD + 512 * w + 16 * q;
        STAMP(2 + 2 * it);
        u32x4 Af[8];
        asm volatile(
            "global_load_dwordx4 %0, %8, off\n\t"
            "global_load_dwordx4 %1, %8, off offset:64\n\t"
            "global_load_dwordx4 %2, %8, off offset:128\n\t"
            "global_load_dwordx4 %3, %8, off offset:192\n\t"
            "global_load_dwordx4 %4, %8, off offset:256\n\t"
            "global_load_dwordx4 %5, %8, off offset:320\n\t"
            "global_load_dwordx4 %6, %8, off offset:384\n\t"
            "global_load_dwordx4 %7, %8, off offset:448\n\t"
            "s_waitcnt vmcnt(0)"
            : "=&v"(Af[0]), "=&v"(Af[1]), "=&v"(Af[2]), "=&v"(Af[3]), "=&v"(Af[4]), "=&v"(Af[5]), "=&v"(Af[6]), "=&v"(Af[7])
            : "v"(src) : "memory");
        f32x4 acc[4];
#pragma unroll
        for (int i = 0; i < 4; ++i) acc[i] = f32x4{0.f, 0.f, 0.f, 0.f};
#pragma unroll
        for (int s = 0; s < 4; ++s) {
            const i32x8 a = join8(Af[2 * s], Af[2 * s + 1]);
#pragma unroll
            for (int i = 0; i < 4; ++i)
                acc[i] = __builtin_amdgcn_mfma_scale_f32_16x16x128_f8f6f4(a, Bf[s][i], acc[i], 0, 0, 0, 0, 0, 0);
        }
        finish(acc, it & 1, p.rank * MS + 16 * lb);
        STAMP(3 + 2 * it);
    }

    unsigned* mk = sm.g.mask[w];
    bool issued = false;
#pragma unroll 1
    for (int k = 0; k < nrem; ++k) {
        const uint8_t* pc; const unsigned* flag; unsigned seq; int row0;
        item(k, pc, flag, seq, row0);
        if (issued) {
            WAITV(2);
        } else {
            if (!bad && polled(k) < seq) {
                const unsigned long long t0 = __builtin_amdgcn_s_memrealtime();
                unsigned itn = 0;
                for (;;) {
                    poll(k, flag);
                    __builtin_amdgcn_s_sleep(3);
                    if (polled(k) >= seq) break;
                    if ((++itn & 15u) == 0u &&
                        (ld_agent(tmo) == call || __builtin_amdgcn_s_memrealtime() - t0 > DEADLINE_TICKS)) {
                        if (l == 0) __hip_atomic_store(tmo, call, __ATOMIC_RELAXED, __HIP_MEMORY_SCOPE_AGENT);
                        bad = true;
                        break;
                    }
                }
            }
            STAMP(2 + 2 * (k + nloc));
            WSTAMP(0);
            fetch_piece(pc, item_plain(k));
            WAITV(0);
        }
        DSTAMP(0);
        WSTAMP(1);
        const uint8_t* pcn; const unsigned* flagn; unsigned seqn; int row0n;
        item(k + 1 < nrem ? k + 1 : k, pcn, flagn, seqn, row0n);
        poll(k + 1 < nrem ? k + 1 : k, flagn);
        f32x4 acc[4];
#pragma unroll
        for (int i2 = 0; i2 < 4; ++i2) acc[i2] = f32x4{0.f, 0.f, 0.f, 0.f};
        if (item_plain(k)) {
            u32x4 fr[8];
#pragma unroll
            for (int s = 0; s < 8; ++s) fr[s] = *(const u32x4*)(ab + s * 256 + l * 4);
            asm volatile("s_waitcnt lgkmcnt(0)" : "+v"(fr[0]), "+v"(fr[1]), "+v"(fr[2]), "+v"(fr[3]), "+v"(fr[4]), "+v"(fr[5]),
                         "+v"(fr[6]), "+v"(fr[7]) :: "memory");
            issued = k + 1 < nrem && !bad && polled(k + 1) >= seqn;
            if (issued) fetch_piece(pcn, item_plain(k + 1));
#pragma unroll
            for (int s = 0; s < 4; ++s) {
                const i32x8 a = join8(fr[2 * s], fr[2 * s + 1]);
#pragma unroll
                for (int i2 = 0; i2 < 4; ++i2)
                    acc[i2] = __builtin_amdgcn_mfma_scale_f32_16x16x128_f8f6f4(a, Bf[s][i2], acc[i2], 0, 0, 0, 0, 0, 0);
            }
        } else {
        *(u32x4*)&mk[l * 4] = u32x4{0u, 0u, 0u, 0u};
        unsigned ex = ab[LIST_OFF / 4 + l];
        u32x4 lo[4];
        u32x3 hi[4];
#pragma unroll
        for (int s = 0; s < 4; ++s) {
            lo[s] = *(const u32x4*)(ab + s * 256 + l * 4);
            const unsigned* h = ab + HI_OFF / 4 + s * 192 + l * 3;
            hi[s] = u32x3{h[0], h[1], h[2]};
        }
        asm volatile("s_waitcnt lgkmcnt(0)" : "+v"(ex), "+v"(lo[0]), "+v"(lo[1]), "+v"(lo[2]), "+v"(lo[3]),
                     "+v"(hi[0]), "+v"(hi[1]), "+v"(hi[2]), "+v"(hi[3]) :: "memory");
        DSTAMP(1);
        issued = k + 1 < nrem && !bad && polled(k + 1) >= seqn;
        if (issued) fetch_piece(pcn, item_plain(k + 1));
        DSTAMP(2);
        const unsigned h0 = (unsigned)__builtin_amdgcn_readlane((int)ex, 0), h1 = (unsigned)__builtin_amdgcn_readlane((int)ex, 1);
        const unsigned nv = h0 & 0xFFFFu, rawb = h0 >> 16;
        const unsigned i0 = 2u * (unsigned)l - 4u;
        if (i0 < nv) atomicOr(&mk[(ex >> 5) & 255u], 1u << (ex & 31u));
        if (i0 + 1u < nv && l >= 2) atomicOr(&mk[(ex >> 21) & 255u], 1u << ((ex >> 16) & 31u));
        unsigned mv[4];
#pragma unroll
        for (int s = 0; s < 4; ++s) mv[s] = mk[s * 64 + l];
        DSTAMP(3);
        if (rawb == 0u) {
            i32x8 a = unpack_unit(lo[0], hi[0], mv[0], (h1 & 0xFFu) * 0x01010101u);
#define SB() __builtin_amdgcn_sched_barrier(0)
#define UF(j) an[j] = (int)(((d[j] & 0xBFBFBFBFu) + A) ^ ((m << (6 - j)) & 0x40404040u))
#define UG(j) gg = and_or(d[j] >> (6 - j), 0x01010101u << j, gg)
#define MM(s, i) acc[i] = __builtin_amdgcn_mfma_scale_f32_16x16x128_f8f6f4(a, Bf[s][i], acc[i], 0, 0, 0, 0, 0, 0)
#pragma unroll
            for (int s = 0; s < 3; ++s) {
                const unsigned d[7] = {lo[s + 1][0], lo[s + 1][1], lo[s + 1][2], lo[s + 1][3], hi[s + 1][0], hi[s + 1][1], hi[s + 1][2]};
                const unsigned m = mv[s + 1], A = ((h1 >> (8 * (s + 1))) & 0xFFu) * 0x01010101u;
                i32x8 an;
                unsigned gg;
                SB(); MM(s, 0); SB();
                UF(0); UF(1); gg = (d[6] << 1) & 0x80808080u; UG(0);
                SB(); MM(s, 1); SB();
                UF(2); UF(3); UG(1); UG(2);
                SB(); MM(s, 2); SB();
                UF(4); UF(5); UG(3); UG(4);
                SB(); MM(s, 3); SB();
                UF(6); UG(5);
                an[7] = (int)((gg + A) ^ ((m >> 1) & 0x40404040u));
                a = an;
            }
            SB(); MM(3, 0); MM(3, 1); MM(3, 2); MM(3, 3); SB();
#undef SB
#undef UF
#undef UG
#undef MM
        } else {
#pragma unroll
            for (int s = 0; s < 4; ++s) {
                i32x8 a;
                if ((rawb >> s) & 1u) {
                    unsigned d7;
                    asm volatile("global_load_dword %0, %1, %2 sc0 sc1\n\ts_waitcnt vmcnt(0)"
                                 : "=&v"(d7) : "v"(RAW7_OFF + s * 256 + l * 4), "s"(pc) : "memory");
                    a[0] = lo[s][0]; a[1] = lo[s][1]; a[2] = lo[s][2]; a[3] = lo[s][3];
                    a[4] = hi[s][0]; a[5] = hi[s][1]; a[6] = hi[s][2]; a[7] = d7;
                } else {
                    a = unpack_unit(lo[s], hi[s], mv[s], ((h1 >> (8 * s)) & 0xFFu) * 0x01010101u);
                }
#pragma unroll
                for (int i2 = 0; i2 < 4; ++i2)
                    acc[i2] = __builtin_amdgcn_mfma_scale_f32_16x16x128_f8f6f4(a, Bf[s][i2], acc[i2], 0, 0, 0, 0, 0, 0);
            }
        }
        }
        DSTAMP(4);
        WSTAMP(2);
        if (!issued && k + 1 < nrem && !bad && polled(k + 1) >= seqn) { fetch_piece(pcn, item_plain(k + 1)); issued = true; }
        {
            const int k2 = k + 2 < nrem ? k + 2 : k;
            const uint8_t* pc2; const unsigned* flag2; unsigned seq2; int row2;
            item(k2, pc2, flag2, seq2, row2);
            poll(k2, flag2);
        }
        finish(acc, (k + nloc) & 1, row0);
        DSTAMP(5);
        STAMP(3 + 2 * (k + nloc));
    }
    STAMP(39);
    STAMP_FLUSH();
}

__global__ void __launch_bounds__(NTHREADS, 2) ag_gemm_kernel(Params p) {
    __shared__ Smem sm;
    const int xcd = blockIdx.x & 7, slot = blockIdx.x >> 3;
    if (slot < 7) pusher(p, sm, slot, xcd);
    else if (slot >= 8) gemm_wg(p, sm, 3 * xcd + ((slot - 8) >> 3), (slot - 8) & 7);
}

}

extern "C" void kernel_launch(int rank, int nranks, ncclComm_t comm, hipStream_t stream,
                              void* const* d_in, const int* in_sizes, int n_in,
                              void* d_out, int out_size, const int* in_dtypes, int out_dtype,
                              void* const* d_out_all, void* const* d_ws_all, size_t ws_bytes) {
    (void)comm; (void)in_dtypes; (void)out_dtype; (void)d_out_all;
    if (nranks != NR || n_in != 4 || (unsigned)rank >= (unsigned)NR || ws_bytes < WS_NEED ||
        in_sizes[0] != MS * KD || in_sizes[1] != KD * NS || out_size != NR * MS * NS) {
        fprintf(stderr, "kernel_launch: unexpected problem shape\n");
        return;
    }
    Params p;
    p.x = (const uint8_t*)d_in[0];
    p.w = (const uint8_t*)d_in[1];
    p.sx = (const float*)d_in[2];
    p.sw = (const float*)d_in[3];
    p.out = (float*)d_out;
    p.me = (uint8_t*)d_ws_all[rank];
    for (int i = 0; i < 7; ++i) p.peer[i] = (uint8_t*)d_ws_all[(rank + 1 + i) % NR];
    p.rank = rank;
    p.pad = 0;
    hipLaunchKernelGGL(ag_gemm_kernel, dim3(NGRID), dim3(NTHREADS), 0, stream, p);
    hipError_t e = hipGetLastError();
    if (e != hipSuccess) fprintf(stderr, "launch error %s\n", hipGetErrorString(e));
}
```

```cpp
#include <hip/hip_runtime.h>
#include <rccl/rccl.h>
#include <cstdio>
#include <cstdint>

typedef unsigned u32x4 __attribute__((ext_vector_type(4)));
typedef unsigned u32x3 __attribute__((ext_vector_type(3)));
typedef int i32x8 __attribute__((ext_vector_type(8)));
typedef float f32x4 __attribute__((ext_vector_type(4)));
typedef float f32x2 __attribute__((ext_vector_type(2)));

namespace {

constexpr int NR = 8;
constexpr int KD = 4096;
constexpr int MS = 512;
constexpr int NS = 512;
constexpr int NBLK = 32;
constexpr int BLK_BYTES = 16 * KD;
constexpr int NGRID = 256;
constexpr int NTHREADS = 512;
constexpr int NGROUP = 24;

constexpr int HI_OFF = 4096, LIST_OFF = 7168, RAW7_OFF = 7424;
constexpr int PIECE_BYTES = 8448;
constexpr int SBLK_BYTES = 8 * PIECE_BYTES;
constexpr int SSHARD_BYTES = NBLK * SBLK_BYTES;
constexpr unsigned LIST_CAP = 124;
constexpr int ABUF_BYTES = 8192;
#define PLAIN_HEAD 1
#define PLAIN_TAIL 2
__host__ __device__ constexpr bool plain_block(int b) { return b < PLAIN_HEAD || b >= NBLK - PLAIN_TAIL; }
constexpr int MAXITEMS = 12;

constexpr size_t CNT_OFF = 0;
constexpr size_t TMO_OFF = 32768;
constexpr size_t FLAG_OFF = 65536;
constexpr size_t DBG_OFF = 131072;
constexpr size_t STAGE_OFF = 1u << 20;
constexpr size_t SLOT_BYTES = (size_t)NR * SSHARD_BYTES;
constexpr size_t WS_NEED = STAGE_OFF + 2 * SLOT_BYTES;

constexpr unsigned long long DEADLINE_TICKS = 40000000ull;

struct Params {
    const uint8_t* x;
    const uint8_t* w;
    const float* sx;
    const float* sw;
    float* out;
    uint8_t* me;
    uint8_t* peer[7];
    int rank;
    int pad;
};

__device__ __forceinline__ unsigned ld_sys(const unsigned* p) {
    return __hip_atomic_load(p, __ATOMIC_RELAXED, __HIP_MEMORY_SCOPE_SYSTEM);
}
__device__ __forceinline__ unsigned ld_agent(const unsigned* p) {
    return __hip_atomic_load(p, __ATOMIC_RELAXED, __HIP_MEMORY_SCOPE_AGENT);
}

#define STAMP(idx) do {} while (0)
#define STAMP_INIT() do {} while (0)
#define STAMP_FLUSH() do {} while (0)

#define DSTAMP(idx) do {} while (0)
#define WSTAMP(idx) do {} while (0)
#define OUTBITS "sc0 sc1"
#define DUR_START() do {} while (0)
#define DUR_END(ws, call) do {} while (0)
#define WAITV(n) asm volatile("s_waitcnt vmcnt(" #n ")" ::: "memory")

__device__ __forceinline__ unsigned draw_call(const Params& p) {
    unsigned* cnt = (unsigned*)(p.me + CNT_OFF + (size_t)blockIdx.x * 128) + (threadIdx.x >> 6);
    unsigned c = 0;
    if ((threadIdx.x & 63) == 0) c = __hip_atomic_fetch_add(cnt, 1u, __ATOMIC_RELAXED, __HIP_MEMORY_SCOPE_AGENT) + 1u;
    return (unsigned)__builtin_amdgcn_readfirstlane((int)c);
}

__device__ __forceinline__ const uint8_t* uniform_ptr(const uint8_t* q) {
    const unsigned long long v = (unsigned long long)q;
    const unsigned lo = (unsigned)__builtin_amdgcn_readfirstlane((int)(unsigned)v);
    const unsigned hi = (unsigned)__builtin_amdgcn_readfirstlane((int)(unsigned)(v >> 32));
    return (const uint8_t*)(((unsigned long long)hi << 32) | lo);
}

struct alignas(16) Smem {
    union {
        struct {
            f32x4 red[2][8][16][16];
            u32x4 abuf[8][ABUF_BYTES / 16];
            unsigned mask[8][256];
            unsigned poll[8][16];
            unsigned bad[2];
        } g;
        struct {
            u32x4 ring[4][4][128];
            unsigned short sub[NBLK][4][32];
            unsigned short meta[NBLK][4];
            unsigned packed[NBLK];
            unsigned done[NBLK];
        } s;
    };
};

__device__ __forceinline__ unsigned and_or(unsigned a, unsigned m, unsigned c) { return (a & m) | c; }
__device__ __forceinline__ unsigned bfi(unsigned m, unsigned a, unsigned b) { return (a & m) | (b & ~m); }

__device__ __forceinline__ unsigned wave_max(unsigned v) {
    v = max(v, (unsigned)__builtin_amdgcn_update_dpp(0, (int)v, 0x111, 0xf, 0xf, true));
    v = max(v, (unsigned)__builtin_amdgcn_update_dpp(0, (int)v, 0x112, 0xf, 0xf, true));
    v = max(v, (unsigned)__builtin_amdgcn_update_dpp(0, (int)v, 0x114, 0xf, 0xf, true));
    v = max(v, (unsigned)__builtin_amdgcn_update_dpp(0, (int)v, 0x118, 0xf, 0xf, true));
    v = max(v, (unsigned)__builtin_amdgcn_update_dpp(0, (int)v, 0x142, 0xa, 0xf, true));
    v = max(v, (unsigned)__builtin_amdgcn_update_dpp(0, (int)v, 0x143, 0xc, 0xf, true));
    return (unsigned)__builtin_amdgcn_readlane((int)v, 63);
}
__device__ __forceinline__ unsigned wave_sum(unsigned v) {
    v += (unsigned)__builtin_amdgcn_update_dpp(0, (int)v, 0x111, 0xf, 0xf, true);
    v += (unsigned)__builtin_amdgcn_update_dpp(0, (int)v, 0x112, 0xf, 0xf, true);
    v += (unsigned)__builtin_amdgcn_update_dpp(0, (int)v, 0x114, 0xf, 0xf, true);
    v += (unsigned)__builtin_amdgcn_update_dpp(0, (int)v, 0x118, 0xf, 0xf, true);
    v += (unsigned)__builtin_amdgcn_update_dpp(0, (int)v, 0x142, 0xa, 0xf, true);
    v += (unsigned)__builtin_amdgcn_update_dpp(0, (int)v, 0x143, 0xc, 0xf, true);
    return (unsigned)__builtin_amdgcn_readlane((int)v, 63);
}

__device__ __forceinline__ void fetch_unit(const uint8_t* src, unsigned lds) {
    unsigned keep;
    asm volatile(
        "s_mov_b32 %0, m0\n\t"
        "s_mov_b32 m0, %3\n\t"
        "s_nop 0\n\t"
        "global_load_lds_dwordx4 %1, off\n\t"
        "s_add_u32 m0, %3, 1024\n\t"
        "s_nop 0\n\t"
        "global_load_lds_dwordx4 %2, off\n\t"
        "s_mov_b32 m0, %0"
        : "=&s"(keep) : "v"(src), "v"(src + 64), "s"(lds) : "memory", "scc");
}

struct Packed {
    u32x4 lo;
    u32x3 hi;
    unsigned d7;
    bool raw;
};

constexpr unsigned SUB_CAP = 31;
__device__ __forceinline__ void pack_unit(const u32x4* slot, Smem& sm, int t, int s, int l, unsigned& a, bool& refresh, Packed& o) {
    const u32x4 R0 = slot[l], R1 = slot[64 + l];
    const unsigned f[8] = {R0[0], R0[1], R0[2], R0[3], R1[0], R1[1], R1[2], R1[3]};
    if (refresh) {
        const unsigned e0 = max(f[0] & 0x7Fu, f[4] & 0x7Fu), e1 = max((f[0] >> 8) & 0x7Fu, (f[4] >> 8) & 0x7Fu);
        const unsigned e2 = max((f[0] >> 16) & 0x7Fu, (f[4] >> 16) & 0x7Fu), e3 = max((f[0] >> 24) & 0x7Fu, (f[4] >> 24) & 0x7Fu);
        const unsigned mx = wave_max(max(max(e0, e1), max(e2, e3)));
        a = mx > 63u ? mx - 63u : 0u;
    }
    const unsigned na = (128u - a) * 0x01010101u;
    unsigned tt[8], m = 0;
#pragma unroll
    for (int j = 0; j < 8; ++j) tt[j] = (f[j] & 0x7F7F7F7Fu) + na;
#pragma unroll
    for (int j = 0; j < 7; ++j) m = and_or(tt[j] >> (6 - j), 0x01010101u << j, m);
    m = and_or(tt[7] << 1, 0x80808080u, m);
    unsigned short* l16 = sm.s.sub[t][s];
    const unsigned tag = (unsigned)((s << 11) | (l << 5));
    unsigned n = 0, mm = m;
    for (;;) {
        const unsigned long long bal = __ballot(mm != 0u);
        if (bal == 0ull || n > SUB_CAP) break;
        const unsigned idx = n + __builtin_amdgcn_mbcnt_hi((unsigned)(bal >> 32), __builtin_amdgcn_mbcnt_lo((unsigned)bal, 0u));
        if (mm != 0u) l16[min(idx, SUB_CAP)] = (unsigned short)(tag | (unsigned)(__ffs(mm) - 1));
        mm &= mm - 1u;
        n += (unsigned)__popcll(bal);
    }
    o.raw = n > SUB_CAP;
    refresh = n > 27u;
    {
        unsigned long long sv;
        const unsigned meta_lds = (unsigned)(uintptr_t)&sm.s.meta[t][s], cnt_lds = (unsigned)(uintptr_t)&sm.s.packed[t];
        asm volatile("s_mov_b64 %0, exec\n\ts_mov_b64 exec, 1\n\tds_write_b16 %1, %2\n\tds_add_u32 %3, %4\n\ts_mov_b64 exec, %0"
                     : "=&s"(sv) : "v"(meta_lds), "v"((a << 8) | (o.raw ? 0xFFu : n)), "v"(cnt_lds), "v"(1u) : "memory");
    }
    const unsigned p7 = and_or(tt[7], 0x3F3F3F3Fu, f[7] & 0x80808080u);
    unsigned d[7];
#pragma unroll
    for (int j = 0; j < 7; ++j) {
        const unsigned sp = j < 6 ? (p7 << (6 - j)) : (p7 >> 1);
        d[j] = and_or(sp, 0x40404040u, and_or(tt[j], 0x3F3F3F3Fu, f[j] & 0x80808080u));
    }
    if (o.raw) {
        o.lo = R0;
        o.hi = u32x3{f[4], f[5], f[6]};
    } else {
        o.lo = u32x4{d[0], d[1], d[2], d[3]};
        o.hi = u32x3{d[4], d[5], d[6]};
    }
    o.d7 = f[7];
}

__device__ __forceinline__ void store_unit(const Packed& o, uint8_t* piece, int s, int l) {
    asm volatile(
        "global_store_dwordx4 %0, %1, %4 sc0 sc1\n\t"
        "global_store_dwordx3 %2, %3, %4 sc0 sc1\n\t"
        "s_nop 1"
        :: "v"(s * 1024 + l * 16), "v"(o.lo), "v"(HI_OFF + s * 768 + l * 12), "v"(o.hi), "s"(piece) : "memory");
    if (o.raw)
        asm volatile("global_store_dword %0, %1, %2 sc0 sc1\n\ts_nop 1"
                     :: "v"(RAW7_OFF + s * 256 + l * 4), "v"(o.d7), "s"(piece) : "memory");
}

__device__ __forceinline__ bool lds_wait(const unsigned* word, unsigned value, unsigned long long t0) {
    unsigned itn = 0;
    while (__hip_atomic_load(word, __ATOMIC_RELAXED, __HIP_MEMORY_SCOPE_WORKGROUP) < value) {
        if ((++itn & 1023u) == 0u && __builtin_amdgcn_s_memrealtime() - t0 > DEADLINE_TICKS) return false;
        __builtin_amdgcn_s_sleep(1);
    }
    asm volatile("" ::: "memory");
    return true;
}

__device__ __forceinline__ void pusher(const Params& p, Smem& sm, int dd, int w) {
    const int tid = threadIdx.x;
    const int s = __builtin_amdgcn_readfirstlane(tid >> 6);
    const int l = tid & 63, r = l & 15, q = l >> 4;
#define WIN 2
    DUR_START();
    STAMP_INIT();
    STAMP(0);
    const uint8_t* const px = p.x;
    uint8_t* const me = p.me;
    uint8_t* const pw = p.peer[dd];
    const int rank = p.rank;
    asm volatile("" :: "s"(px), "s"(me), "s"(pw), "s"(rank));
    const uint8_t* src = px + (size_t)r * KD + 512 * w + 128 * (s & 3) + 16 * q;
    u32x4* ring = &sm.s.ring[s & 3][0][0];
    const unsigned ring_lds = (unsigned)__builtin_amdgcn_readfirstlane((int)(unsigned)(uintptr_t)ring);
    if (__builtin_expect(s < 4, 1)) {
        fetch_unit(src, ring_lds);
        fetch_unit(src + BLK_BYTES, ring_lds + 2048u);
        if (WIN == 3) fetch_unit(src + 2 * BLK_BYTES, ring_lds + 4096u);
    }
    unsigned ticket;
    {
        unsigned long long sv;
        asm volatile("s_mov_b64 %1, exec\n\ts_mov_b64 exec, 1\n\tglobal_atomic_add %0, %2, %3, off sc0\n\ts_mov_b64 exec, %1"
                     : "=&v"(ticket), "=&s"(sv)
                     : "v"((unsigned*)(me + CNT_OFF + (size_t)blockIdx.x * 128) + s), "v"(1u) : "memory");
    }
    if (tid < NBLK) { sm.s.packed[tid] = 0u; sm.s.done[tid] = 0u; }
    asm volatile("s_waitcnt lgkmcnt(0)" ::: "memory");
    __builtin_amdgcn_s_barrier();
    asm volatile("s_waitcnt vmcnt(0)" : "+v"(ticket) :: "memory");
    if (s > 4) return;
    const unsigned call = (unsigned)__builtin_amdgcn_readfirstlane((int)ticket) + 1u;
    if (s == 4) {
        unsigned* flag = (unsigned*)(pw + FLAG_OFF + (size_t)(rank * 8 + w) * 128);
        const unsigned base = (call - 1u) * NBLK;
        uint8_t* dst = (uint8_t*)uniform_ptr(pw + STAGE_OFF + (size_t)(call & 1u) * SLOT_BYTES + (size_t)rank * SSHARD_BYTES +
                                             w * PIECE_BYTES + LIST_OFF);
        const unsigned long long t0 = __builtin_amdgcn_s_memrealtime();
#define PUBLISH(t) do { unsigned long long sv_; \
            asm volatile("s_mov_b64 %0, exec\n\ts_mov_b64 exec, 1\n\tglobal_atomic_umax %1, %2, off sc1\n\ts_mov_b64 exec, %0" \
                         : "=&s"(sv_) : "v"(flag), "v"(base + (unsigned)(t) + 1u) : "memory"); } while (0)
#pragma unroll 1
        for (int t = 0; t < NBLK; ++t) {
            if (!lds_wait(&sm.s.packed[t], 4u, t0)) return;
            const bool plain = plain_block(t);
            unsigned v = 0, nv = 0;
            if (!plain) {
                const unsigned m01 = (unsigned)__builtin_amdgcn_readfirstlane((int)((const unsigned*)sm.s.meta[t])[0]);
                const unsigned m23 = (unsigned)__builtin_amdgcn_readfirstlane((int)((const unsigned*)sm.s.meta[t])[1]);
                const unsigned c0 = m01 & 0xFFu, c1 = (m01 >> 16) & 0xFFu, c2 = m23 & 0xFFu, c3 = (m23 >> 16) & 0xFFu;
                const unsigned rawb = (c0 == 0xFFu ? 1u : 0u) | (c1 == 0xFFu ? 2u : 0u) | (c2 == 0xFFu ? 4u : 0u) | (c3 == 0xFFu ? 8u : 0u);
                const unsigned p1 = c0 == 0xFFu ? 0u : c0, p2 = p1 + (c1 == 0xFFu ? 0u : c1), p3 = p2 + (c2 == 0xFFu ? 0u : c2);
                nv = p3 + (c3 == 0xFFu ? 0u : c3);
                const unsigned short* sub = &sm.s.sub[t][0][0];
#pragma unroll
                for (int h = 0; h < 2; ++h) {
                    const unsigned i = 2u * (unsigned)l - 4u + (unsigned)h;
                    unsigned d = i >= p1 ? 32u - p1 : 0u;
                    d = i >= p2 ? 64u - p2 : d;
                    d = i >= p3 ? 96u - p3 : d;
                    v |= (unsigned)sub[(i + d) & 127u] << (16 * h);
                }
                if (l == 0) v = nv | (rawb << 16);
                if (l == 1) v = ((m01 >> 8) & 0xFFu) | ((m01 >> 24) << 8) | (((m23 >> 8) & 0xFFu) << 16) | ((m23 >> 24) << 24);
            }
            const unsigned nl = plain ? 1u : 2u + (nv + 1u) / 2u;
            const unsigned long long em = nl >= 64u ? ~0ull : ((1ull << nl) - 1ull);
            unsigned long long sv;
            asm volatile("s_mov_b64 %0, exec\n\ts_mov_b64 exec, %4\n\tglobal_store_dword %1, %2, %3 sc0 sc1\n\ts_mov_b64 exec, %0"
                         : "=&s"(sv) : "v"(l * 4 + (plain ? PIECE_BYTES - LIST_OFF - 4 : 0)), "v"(v), "s"(dst + (size_t)t * SBLK_BYTES), "s"(em) : "memory");
            if (t >= 2) {
                if (!lds_wait(&sm.s.done[t - 2], 4u, t0)) return;
                if (t >= 4) WAITV(4); else WAITV(2);
                PUBLISH(t - 2);
            }
        }
        if (!lds_wait(&sm.s.done[NBLK - 2], 4u, t0)) return;
        WAITV(3);
        PUBLISH(NBLK - 2);
        if (!lds_wait(&sm.s.done[NBLK - 1], 4u, t0)) return;
        WAITV(2);
        PUBLISH(NBLK - 1);
#undef PUBLISH
        return;
    }
#define PACE 1656
    uint8_t* dst = (uint8_t*)uniform_ptr(pw + STAGE_OFF + (size_t)(call & 1u) * SLOT_BYTES + (size_t)rank * SSHARD_BYTES +
                                         w * PIECE_BYTES);
    STAMP(1);
    const unsigned long long tstart = __builtin_amdgcn_s_memrealtime();
    unsigned win_a = 0;
    bool win_refresh = true;
#define LDS_COUNT(word) do { unsigned long long sv_; \
        asm volatile("s_mov_b64 %0, exec\n\ts_mov_b64 exec, 1\n\tds_add_u32 %1, %2\n\ts_mov_b64 exec, %0" \
                     : "=&s"(sv_) : "v"((unsigned)(uintptr_t)&(word)), "v"(1u) : "memory"); } while (0)
#define SIGNAL(t) LDS_COUNT(sm.s.done[t])
#pragma unroll 1
    for (int t = 0; t < NBLK; ++t) {
        if (WIN == 2) { if (t >= 2) WAITV(6); }
        else { if (t >= 3) WAITV(10); }
        Packed o;
        const bool plain = plain_block(t);
        u32x4 r0, r1;
        if (plain) {
            r0 = ring[(t & 3) * 128 + l];
            r1 = ring[(t & 3) * 128 + 64 + l];
            asm volatile("s_waitcnt lgkmcnt(0)" : "+v"(r0), "+v"(r1) :: "memory");
            LDS_COUNT(sm.s.packed[t]);
        } else {
            pack_unit(ring + (t & 3) * 128, sm, t, s, l, win_a, win_refresh, o);
        }
        const int tn = t + WIN < NBLK ? t + WIN : t;
        fetch_unit(src + (size_t)tn * BLK_BYTES, ring_lds + (unsigned)((t + WIN) & 3) * 2048u);
        if (WIN == 2) { if (t >= 2) { WAITV(6); SIGNAL(t - 2); if (t == 2) STAMP(4); if (t == 3) STAMP(5); if (t == 6) STAMP(6); } }
        else { if (t >= 3) { WAITV(10); SIGNAL(t - 3); } }
        if (PACE > 0) {
            const int nplain = (t < PLAIN_HEAD ? t : PLAIN_HEAD) + (t > NBLK - PLAIN_TAIL ? t - (NBLK - PLAIN_TAIL) : 0);
            const unsigned long long due = tstart + (((unsigned long long)(8 * t + nplain) * PACE) >> 7);
            while (__builtin_amdgcn_s_memrealtime() < due) __builtin_amdgcn_s_sleep(1);
        }
        if (plain)
            asm volatile(
                "global_store_dwordx4 %0, %1, %3 sc0 sc1\n\t"
                "global_store_dwordx4 %0, %2, %3 offset:1024 sc0 sc1\n\t"
                "s_nop 1"
                :: "v"(s * 2048 + l * 16), "v"(r0), "v"(r1), "s"(dst + (size_t)t * SBLK_BYTES) : "memory");
        else
            store_unit(o, dst + (size_t)t * SBLK_BYTES, s, l);
        if (t == 15) STAMP(2);
    }
    if (WIN == 3) { WAITV(8); SIGNAL(NBLK - 3); }
    WAITV(4); SIGNAL(NBLK - 2);
    WAITV(0); SIGNAL(NBLK - 1);
    STAMP(3);
    DUR_END(me, call);
#undef SIGNAL
#undef LDS_COUNT
}

__device__ __forceinline__ void transpose4(unsigned a, unsigned b, unsigned c, unsigned d,
                                           unsigned& o0, unsigned& o1, unsigned& o2, unsigned& o3) {
    const unsigned t0 = __builtin_amdgcn_perm(b, a, 0x05010400u);
    const unsigned t1 = __builtin_amdgcn_perm(b, a, 0x07030602u);
    const unsigned t2 = __builtin_amdgcn_perm(d, c, 0x05010400u);
    const unsigned t3 = __builtin_amdgcn_perm(d, c, 0x07030602u);
    o0 = __builtin_amdgcn_perm(t2, t0, 0x05040100u);
    o1 = __builtin_amdgcn_perm(t2, t0, 0x07060302u);
    o2 = __builtin_amdgcn_perm(t3, t1, 0x05040100u);
    o3 = __builtin_amdgcn_perm(t3, t1, 0x07060302u);
}

__device__ __forceinline__ i32x8 join8(u32x4 lo, u32x4 hi) {
    i32x8 v;
    v[0] = lo[0]; v[1] = lo[1]; v[2] = lo[2]; v[3] = lo[3];
    v[4] = hi[0]; v[5] = hi[1]; v[6] = hi[2]; v[7] = hi[3];
    return v;
}

__device__ __forceinline__ i32x8 unpack_unit(u32x4 lo, u32x3 hi, unsigned m, unsigned A) {
    const unsigned d[7] = {lo[0], lo[1], lo[2], lo[3], hi[0], hi[1], hi[2]};
    i32x8 f;
    unsigned g = (d[6] << 1) & 0x80808080u;
#pragma unroll
    for (int j = 0; j < 6; ++j) g = and_or(d[j] >> (6 - j), 0x01010101u << j, g);
#pragma unroll
    for (int j = 0; j < 7; ++j) f[j] = (int)(((d[j] & 0xBFBFBFBFu) + A) ^ ((m << (6 - j)) & 0x40404040u));
    f[7] = (int)((g + A) ^ ((m >> 1) & 0x40404040u));
    return f;
}

__device__ __forceinline__ void gemm_wg(const Params& p, Smem& sm, int g, int j) {
    const int tid = threadIdx.x, w = __builtin_amdgcn_readfirstlane(tid >> 6), l = tid & 63, r = l & 15, q = l >> 4;
    DUR_START();
    STAMP_INIT();
    STAMP(0);
    if (tid < 2) sm.g.bad[tid] = 0u;
    if (tid < 128) (&sm.g.poll[0][0])[tid] = 0u;
#define WDELAY 8
    for (int i = 0; i < WDELAY; ++i) __builtin_amdgcn_s_sleep(2);
    i32x8 Bf[4][4];
    {
        const uint8_t* wp = p.w + (size_t)(512 * w + 16 * q) * NS + 64 * j + 4 * r;
#pragma unroll
        for (int s = 0; s < 4; ++s) {
            unsigned t[32];
#pragma unroll
            for (int h = 0; h < 2; ++h)
#pragma unroll
                for (int k = 0; k < 16; ++k)
                    t[h * 16 + k] = *(const unsigned*)(wp + (size_t)(128 * s + 64 * h + k) * NS);
#pragma unroll
            for (int jj = 0; jj < 8; ++jj) {
                unsigned o0, o1, o2, o3;
                transpose4(t[4 * jj], t[4 * jj + 1], t[4 * jj + 2], t[4 * jj + 3], o0, o1, o2, o3);
                Bf[s][0][jj] = o0; Bf[s][1][jj] = o1; Bf[s][2][jj] = o2; Bf[s][3][jj] = o3;
            }
        }
    }
    const float scale = p.sx[0] * p.sw[0];
    const unsigned call = draw_call(p);
    __syncthreads();
    STAMP(1);
    unsigned* tmo = (unsigned*)(p.me + TMO_OFF);
    const uint8_t* stage = p.me + STAGE_OFF + (size_t)(call & 1u) * SLOT_BYTES + w * PIECE_BYTES;
    float* outp = p.out + (size_t)(tid >> 5) * NS + 64 * j + (tid & 31) * 2;
    const int nloc = g >= 16 ? 2 : 1;
    const int nrem = g < 8 ? 10 : 9;
    bool bad = false;

    auto finish = [&](const f32x4 (&acc)[4], int buf, int row0) {
#pragma unroll
        for (int t = 0; t < 4; ++t) {
            f32x4 v = f32x4{acc[0][t], acc[1][t], acc[2][t], acc[3][t]};
            asm volatile("" : "+v"(v));
            sm.g.red[buf][w][4 * q + t][r] = v;
        }
        if (bad) sm.g.bad[buf] = 1u;
        __syncthreads();
        const bool anybad = sm.g.bad[buf] != 0u;
        const f32x2* rp = (const f32x2*)&sm.g.red[buf][0][0][0] + tid;
        f32x2 v = rp[0];
#pragma unroll
        for (int k = 1; k < 8; ++k) v += rp[k * 512];
        v *= scale;
        v[0] = fmaxf(v[0], 0.f);
        v[1] = fmaxf(v[1], 0.f);
        if (anybad) v = f32x2{__builtin_nanf(""), __builtin_nanf("")};
        asm volatile("global_store_dwordx2 %0, %1, off " OUTBITS "\n\ts_nop 0" :: "v"(outp + (size_t)row0 * NS), "v"(v) : "memory");
    };

    auto item_plain = [&](int k) -> bool { return plain_block((g + NGROUP * k) / 7); };
    auto item = [&](int k, const uint8_t*& pc, const unsigned*& flag, unsigned& seq, int& row0) {
        const int n = g + NGROUP * k;
        const int i = n % 7, b = n / 7;
        const unsigned srcr = (unsigned)(p.rank + 1 + i) & 7u;
        pc = uniform_ptr(stage + (size_t)srcr * SSHARD_BYTES + (size_t)b * SBLK_BYTES);
        flag = (const unsigned*)uniform_ptr(p.me + FLAG_OFF + (size_t)(srcr * 8 + w) * 128);
        seq = (call - 1u) * NBLK + b + 1u;
        row0 = (int)srcr * MS + 16 * b;
    };
    const unsigned poll_lds = (unsigned)__builtin_amdgcn_readfirstlane((int)(unsigned)(uintptr_t)&sm.g.poll[w][0]);
    auto poll = [&](int k, const unsigned* flag) {
        unsigned keep; unsigned long long sv;
        asm volatile(
            "s_mov_b32 %0, m0\n\ts_mov_b64 %1, exec\n\ts_mov_b32 m0, %4\n\ts_mov_b64 exec, 1\n\t"
            "global_load_lds_dword %2, %3 sc0 sc1\n\t"
            "s_mov_b64 exec, %1\n\ts_mov_b32 m0, %0"
            : "=&s"(keep), "=&s"(sv) : "v"(0u), "s"(flag), "s"(poll_lds + 4u * (unsigned)k) : "memory");
    };
    auto polled = [&](int k) -> unsigned {
        return (unsigned)__builtin_amdgcn_readfirstlane((int)__hip_atomic_load(&sm.g.poll[w][k], __ATOMIC_RELAXED, __HIP_MEMORY_SCOPE_WORKGROUP));
    };
    const unsigned* ab = (const unsigned*)&sm.g.abuf[w][0];
    const unsigned ab_lds = (unsigned)__builtin_amdgcn_readfirstlane((int)(unsigned)(uintptr_t)ab);
    auto fetch_piece = [&](const uint8_t* pc, bool plain) {
        unsigned keep; unsigned long long sv;
        asm volatile(
            "s_mov_b32 %0, m0\n\t"
            "s_mov_b32 m0, %5\n\ts_nop 0\n\t"
            "global_load_lds_dwordx4 %2, %3 sc0 sc1\n\t"
            "global_load_lds_dwordx4 %2, %3 offset:1024 sc0 sc1\n\t"
            "global_load_lds_dwordx4 %2, %3 offset:2048 sc0 sc1\n\t"
            "global_load_lds_dwordx4 %2, %3 offset:3072 sc0 sc1\n\t"
            "s_add_u32 m0, %5, 4096\n\ts_nop 0\n\t"
            "global_load_lds_dwordx4 %2, %4 sc0 sc1\n\t"
            "global_load_lds_dwordx4 %2, %4 offset:1024 sc0 sc1\n\t"
            "global_load_lds_dwordx4 %2, %4 offset:2048 sc0 sc1\n\t"
            "s_mov_b64 %1, exec\n\ts_mov_b64 exec, %6\n\t"
            "global_load_lds_dwordx4 %2, %4 offset:3072 sc0 sc1\n\t"
            "s_mov_b64 exec, %1\n\t"
            "s_mov_b32 m0, %0"
            : "=&s"(keep), "=&s"(sv) : "v"(l * 16), "s"(pc), "s"(pc + 4096), "s"(ab_lds), "s"(plain ? ~0ull : 0xffffull) : "memory", "scc");
    };
    {
        const uint8_t* pc; const unsigned* flag; unsigned seq; int row0;
        item(0, pc, flag, seq, row0); poll(0, flag);
        item(1, pc, flag, seq, row0); poll(1, flag);
    }

    for (int it = 0; it < nloc; ++it) {
        const int lb = g + 8 * it;
        const uint8_t* src = p.x + (size_t)(16 * lb + r) * KD + 512 * w + 16 * q;
        STAMP(2 + 2 * it);
        u32x4 Af[8];
        asm volatile(
            "global_load_dwordx4 %0, %8, off\n\t"
            "global_load_dwordx4 %1, %8, off offset:64\n\t"
            "global_load_dwordx4 %2, %8, off offset:128\n\t"
            "global_load_dwordx4 %3, %8, off offset:192\n\t"
            "global_load_dwordx4 %4, %8, off offset:256\n\t"
            "global_load_dwordx4 %5, %8, off offset:320\n\t"
            "global_load_dwordx4 %6, %8, off offset:384\n\t"
            "global_load_dwordx4 %7, %8, off offset:448\n\t"
            "s_waitcnt vmcnt(0)"
            : "=&v"(Af[0]), "=&v"(Af[1]), "=&v"(Af[2]), "=&v"(Af[3]), "=&v"(Af[4]), "=&v"(Af[5]), "=&v"(Af[6]), "=&v"(Af[7])
            : "v"(src) : "memory");
        f32x4 acc[4];
#pragma unroll
        for (int i = 0; i < 4; ++i) acc[i] = f32x4{0.f, 0.f, 0.f, 0.f};
#pragma unroll
        for (int s = 0; s < 4; ++s) {
            const i32x8 a = join8(Af[2 * s], Af[2 * s + 1]);
#pragma unroll
            for (int i = 0; i < 4; ++i)
                acc[i] = __builtin_amdgcn_mfma_scale_f32_16x16x128_f8f6f4(a, Bf[s][i], acc[i], 0, 0, 0, 0, 0, 0);
        }
        finish(acc, it & 1, p.rank * MS + 16 * lb);
        STAMP(3 + 2 * it);
    }

    unsigned* mk = sm.g.mask[w];
    bool issued = false;
#pragma unroll 1
    for (int k = 0; k < nrem; ++k) {
        const uint8_t* pc; const unsigned* flag; unsigned seq; int row0;
        item(k, pc, flag, seq, row0);
        const bool was_issued = issued;
        if (issued) {
            WAITV(2);
        } else {
            if (!bad && polled(k) < seq) {
                const unsigned long long t0 = __builtin_amdgcn_s_memrealtime();
                unsigned itn = 0;
                for (;;) {
                    poll(k, flag);
                    __builtin_amdgcn_s_sleep(3);
                    if (polled(k) >= seq) break;
                    if ((++itn & 15u) == 0u &&
                        (ld_agent(tmo) == call || __builtin_amdgcn_s_memrealtime() - t0 > DEADLINE_TICKS)) {
                        if (l == 0) __hip_atomic_store(tmo, call, __ATOMIC_RELAXED, __HIP_MEMORY_SCOPE_AGENT);
                        bad = true;
                        break;
                    }
                }
            }
            STAMP(2 + 2 * (k + nloc));
            WSTAMP(0);
            if (!item_plain(k)) {
                fetch_piece(pc, false);
                WAITV(0);
            }
        }
        const bool direct = !was_issued && item_plain(k);
        DSTAMP(0);
        WSTAMP(1);
        const bool last = k + 1 == nrem;
        const uint8_t* pcn = pc; const unsigned* flagn = flag; unsigned seqn = 0xFFFFFFFFu; int row0n;
        if (!last) {
            item(k + 1, pcn, flagn, seqn, row0n);
            poll(k + 1, flagn);
        }
        f32x4 acc[4];
#pragma unroll
        for (int i2 = 0; i2 < 4; ++i2) acc[i2] = f32x4{0.f, 0.f, 0.f, 0.f};
        if (direct) {
            u32x4 fr[8];
            const uint8_t* pc4 = pc + 4096;
            asm volatile(
                "global_load_dwordx4 %0, %8, %9 sc0 sc1\n\t"
                "global_load_dwordx4 %1, %8, %9 offset:1024 sc0 sc1\n\t"
                "global_load_dwordx4 %2, %8, %9 offset:2048 sc0 sc1\n\t"
                "global_load_dwordx4 %3, %8, %9 offset:3072 sc0 sc1\n\t"
                "global_load_dwordx4 %4, %8, %10 sc0 sc1\n\t"
                "global_load_dwordx4 %5, %8, %10 offset:1024 sc0 sc1\n\t"
                "global_load_dwordx4 %6, %8, %10 offset:2048 sc0 sc1\n\t"
                "global_load_dwordx4 %7, %8, %10 offset:3072 sc0 sc1"
                : "=&v"(fr[0]), "=&v"(fr[1]), "=&v"(fr[2]), "=&v"(fr[3]), "=&v"(fr[4]), "=&v"(fr[5]), "=&v"(fr[6]), "=&v"(fr[7])
                : "v"(l * 16), "s"(pc), "s"(pc4) : "memory");
#define KSTEP(s, nw) { \
            asm volatile("s_waitcnt vmcnt(" #nw ")" : "+v"(fr[2 * s]), "+v"(fr[2 * s + 1]) :: "memory"); \
            const i32x8 a = join8(fr[2 * s], fr[2 * s + 1]); \
            _Pragma("unroll") for (int i2 = 0; i2 < 4; ++i2) \
                acc[i2] = __builtin_amdgcn_mfma_scale_f32_16x16x128_f8f6f4(a, Bf[s][i2], acc[i2], 0, 0, 0, 0, 0, 0); \
            __builtin_amdgcn_sched_barrier(0); }
            KSTEP(0, 6) KSTEP(1, 4) KSTEP(2, 2) KSTEP(3, 0)
#undef KSTEP
            issued = false;
        } else if (item_plain(k)) {
            u32x4 fr[8];
#pragma unroll
            for (int s = 0; s < 8; ++s) fr[s] = *(const u32x4*)(ab + s * 256 + l * 4);
            asm volatile("s_waitcnt lgkmcnt(0)" : "+v"(fr[0]), "+v"(fr[1]), "+v"(fr[2]), "+v"(fr[3]), "+v"(fr[4]), "+v"(fr[5]),
                         "+v"(fr[6]), "+v"(fr[7]) :: "memory");
            issued = !last && !bad && polled(k + 1) >= seqn;
            if (issued) fetch_piece(pcn, item_plain(k + 1));
#pragma unroll
            for (int s = 0; s < 4; ++s) {
                const i32x8 a = join8(fr[2 * s], fr[2 * s + 1]);
#pragma unroll
                for (int i2 = 0; i2 < 4; ++i2)
                    acc[i2] = __builtin_amdgcn_mfma_scale_f32_16x16x128_f8f6f4(a, Bf[s][i2], acc[i2], 0, 0, 0, 0, 0, 0);
            }
        } else {
        *(u32x4*)&mk[l * 4] = u32x4{0u, 0u, 0u, 0u};
        unsigned ex = ab[LIST_OFF / 4 + l];
        u32x4 lo[4];
        u32x3 hi[4];
#pragma unroll
        for (int s = 0; s < 4; ++s) {
            lo[s] = *(const u32x4*)(ab + s * 256 + l * 4);
            const unsigned* h = ab + HI_OFF / 4 + s * 192 + l * 3;
            hi[s] = u32x3{h[0], h[1], h[2]};
        }
        asm volatile("s_waitcnt lgkmcnt(0)" : "+v"(ex), "+v"(lo[0]), "+v"(lo[1]), "+v"(lo[2]), "+v"(lo[3]),
                     "+v"(hi[0]), "+v"(hi[1]), "+v"(hi[2]), "+v"(hi[3]) :: "memory");
        DSTAMP(1);
        issued = !last && !bad && polled(k + 1) >= seqn;
        if (issued) fetch_piece(pcn, item_plain(k + 1));
        DSTAMP(2);
        const unsigned h0 = (unsigned)__builtin_amdgcn_readlane((int)ex, 0), h1 = (unsigned)__builtin_amdgcn_readlane((int)ex, 1);
        const unsigned nv = h0 & 0xFFFFu, rawb = h0 >> 16;
        const unsigned i0 = 2u * (unsigned)l - 4u;
        if (i0 < nv) atomicOr(&mk[(ex >> 5) & 255u], 1u << (ex & 31u));
        if (i0 + 1u < nv && l >= 2) atomicOr(&mk[(ex >> 21) & 255u], 1u << ((ex >> 16) & 31u));
        unsigned mv[4];
#pragma unroll
        for (int s = 0; s < 4; ++s) mv[s] = mk[s * 64 + l];
        DSTAMP(3);
        if (rawb == 0u) {
            i32x8 a = unpack_unit(lo[0], hi[0], mv[0], (h1 & 0xFFu) * 0x01010101u);
#define SB() __builtin_amdgcn_sched_barrier(0)
#define UF(j) an[j] = (int)(((d[j] & 0xBFBFBFBFu) + A) ^ ((m << (6 - j)) & 0x40404040u))
#define UG(j) gg = and_or(d[j] >> (6 - j), 0x01010101u << j, gg)
#define MM(s, i) acc[i] = __builtin_amdgcn_mfma_scale_f32_16x16x128_f8f6f4(a, Bf[s][i], acc[i], 0, 0, 0, 0, 0, 0)
#pragma unroll
            for (int s = 0; s < 3; ++s) {
                const unsigned d[7] = {lo[s + 1][0], lo[s + 1][1], lo[s + 1][2], lo[s + 1][3], hi[s + 1][0], hi[s + 1][1], hi[s + 1][2]};
                const unsigned m = mv[s + 1], A = ((h1 >> (8 * (s + 1))) & 0xFFu) * 0x01010101u;
                i32x8 an;
                unsigned gg;
                SB(); MM(s, 0); SB();
                UF(0); UF(1); gg = (d[6] << 1) & 0x80808080u; UG(0);
                SB(); MM(s, 1); SB();
                UF(2); UF(3); UG(1); UG(2);
                SB(); MM(s, 2); SB();
                UF(4); UF(5); UG(3); UG(4);
                SB(); MM(s, 3); SB();
                UF(6); UG(5);
                an[7] = (int)((gg + A) ^ ((m >> 1) & 0x40404040u));
                a = an;
            }
            SB(); MM(3, 0); MM(3, 1); MM(3, 2); MM(3, 3); SB();
#undef SB
#undef UF
#undef UG
#undef MM
        } else {
#pragma unroll
            for (int s = 0; s < 4; ++s) {
                i32x8 a;
                if ((rawb >> s) & 1u) {
                    unsigned d7;
                    asm volatile("global_load_dword %0, %1, %2 sc0 sc1\n\ts_waitcnt vmcnt(0)"
                                 : "=&v"(d7) : "v"(RAW7_OFF + s * 256 + l * 4), "s"(pc) : "memory");
                    a[0] = lo[s][0]; a[1] = lo[s][1]; a[2] = lo[s][2]; a[3] = lo[s][3];
                    a[4] = hi[s][0]; a[5] = hi[s][1]; a[6] = hi[s][2]; a[7] = d7;
                } else {
                    a = unpack_unit(lo[s], hi[s], mv[s], ((h1 >> (8 * s)) & 0xFFu) * 0x01010101u);
                }
#pragma unroll
                for (int i2 = 0; i2 < 4; ++i2)
                    acc[i2] = __builtin_amdgcn_mfma_scale_f32_16x16x128_f8f6f4(a, Bf[s][i2], acc[i2], 0, 0, 0, 0, 0, 0);
            }
        }
        }
        DSTAMP(4);
        WSTAMP(2);
        if (!last) {
            if (!issued && !bad && polled(k + 1) >= seqn) { fetch_piece(pcn, item_plain(k + 1)); issued = true; }
            const int k2 = k + 2 < nrem ? k + 2 : k;
            const uint8_t* pc2; const unsigned* flag2; unsigned seq2; int row2;
            item(k2, pc2, flag2, seq2, row2);
            poll(k2, flag2);
        }
        finish(acc, (k + nloc) & 1, row0);
        DSTAMP(5);
        STAMP(3 + 2 * (k + nloc));
    }
    DUR_END(p.me, call);
    STAMP(39);
    STAMP_FLUSH();
}

__global__ void __launch_bounds__(NTHREADS, 2) ag_gemm_kernel(Params p) {
    __shared__ Smem sm;
    const int xcd = blockIdx.x & 7, slot = blockIdx.x >> 3;
    if (__builtin_expect(slot < 7, 1)) pusher(p, sm, slot, xcd);
    else if (slot >= 8) gemm_wg(p, sm, 3 * xcd + ((slot - 8) >> 3), (slot - 8) & 7);
}

}

extern "C" void kernel_launch(int rank, int nranks, ncclComm_t comm, hipStream_t stream,
                              void* const* d_in, const int* in_sizes, int n_in,
                              void* d_out, int out_size, const int* in_dtypes, int out_dtype,
                              void* const* d_out_all, void* const* d_ws_all, size_t ws_bytes) {
    (void)comm; (void)in_dtypes; (void)out_dtype; (void)d_out_all;
    if (nranks != NR || n_in != 4 || (unsigned)rank >= (unsigned)NR || ws_bytes < WS_NEED ||
        in_sizes[0] != MS * KD || in_sizes[1] != KD * NS || out_size != NR * MS * NS) {
        fprintf(stderr, "kernel_launch: unexpected problem shape\n");
        return;
    }
    Params p;
    p.x = (const uint8_t*)d_in[0];
    p.w = (const uint8_t*)d_in[1];
    p.sx = (const float*)d_in[2];
    p.sw = (const float*)d_in[3];
    p.out = (float*)d_out;
    p.me = (uint8_t*)d_ws_all[rank];
    for (int i = 0; i < 7; ++i) p.peer[i] = (uint8_t*)d_ws_all[(rank + 1 + i) % NR];
    p.rank = rank;
    p.pad = 0;
    hipLaunchKernelGGL(ag_gemm_kernel, dim3(NGRID), dim3(NTHREADS), 0, stream, p);
    hipError_t e = hipGetLastError();
    if (e != hipSuccess) fprintf(stderr, "launch error %s\n", hipGetErrorString(e));
}
```

```cpp
#include <hip/hip_runtime.h>
#include <rccl/rccl.h>
#include <cstdio>
#include <cstdint>

typedef unsigned u32x4 __attribute__((ext_vector_type(4)));
typedef unsigned u32x3 __attribute__((ext_vector_type(3)));
typedef int i32x8 __attribute__((ext_vector_type(8)));
typedef float f32x4 __attribute__((ext_vector_type(4)));
typedef float f32x2 __attribute__((ext_vector_type(2)));

namespace {

constexpr int NR = 8;
constexpr int KD = 4096;
constexpr int MS = 512;
constexpr int NS = 512;
constexpr int NBLK = 32;
constexpr int BLK_BYTES = 16 * KD;
constexpr int NGRID = 256;
constexpr int NTHREADS = 512;
constexpr int NGROUP = 24;

constexpr int HI_OFF = 4096, LIST_OFF = 7168, RAW7_OFF = 7424;
constexpr int PIECE_BYTES = 8448;
constexpr int SBLK_BYTES = 8 * PIECE_BYTES;
constexpr int SSHARD_BYTES = NBLK * SBLK_BYTES;
constexpr int ABUF_BYTES = 8192;
constexpr int PLAIN_HEAD = 1, PLAIN_TAIL = 2;
__host__ __device__ constexpr bool plain_block(int b) { return b < PLAIN_HEAD || b >= NBLK - PLAIN_TAIL; }
constexpr unsigned PACE = 1652;

constexpr size_t CNT_OFF = 0;
constexpr size_t TMO_OFF = 32768;
constexpr size_t T0_OFF = 36864;
constexpr size_t FLAG_OFF = 65536;
constexpr size_t STAGE_OFF = 1u << 20;
constexpr size_t SLOT_BYTES = (size_t)NR * SSHARD_BYTES;
constexpr size_t WS_NEED = STAGE_OFF + 2 * SLOT_BYTES;

constexpr unsigned long long DEADLINE_TICKS = 40000000ull;

struct Params {
    const uint8_t* x;
    const uint8_t* w;
    const float* sx;
    const float* sw;
    float* out;
    uint8_t* me;
    uint8_t* peer[7];
    int rank;
    int pad;
};

__device__ __forceinline__ unsigned ld_agent(const unsigned* p) {
    return __hip_atomic_load(p, __ATOMIC_RELAXED, __HIP_MEMORY_SCOPE_AGENT);
}

#define WAITV(n) asm volatile("s_waitcnt vmcnt(" #n ")" ::: "memory")

__device__ __forceinline__ unsigned draw_call(const Params& p) {
    unsigned* cnt = (unsigned*)(p.me + CNT_OFF + (size_t)blockIdx.x * 128) + (threadIdx.x >> 6);
    unsigned c = 0;
    if ((threadIdx.x & 63) == 0) c = __hip_atomic_fetch_add(cnt, 1u, __ATOMIC_RELAXED, __HIP_MEMORY_SCOPE_AGENT) + 1u;
    return (unsigned)__builtin_amdgcn_readfirstlane((int)c);
}

__device__ __forceinline__ const uint8_t* uniform_ptr(const uint8_t* q) {
    const unsigned long long v = (unsigned long long)q;
    const unsigned lo = (unsigned)__builtin_amdgcn_readfirstlane((int)(unsigned)v);
    const unsigned hi = (unsigned)__builtin_amdgcn_readfirstlane((int)(unsigned)(v >> 32));
    return (const uint8_t*)(((unsigned long long)hi << 32) | lo);
}

struct alignas(16) Smem {
    union {
        struct {
            f32x4 red[2][8][16][16];
            u32x4 abuf[8][ABUF_BYTES / 16];
            unsigned mask[8][256];
            unsigned poll[8][16];
            unsigned bad[2];
        } g;
        struct {
            u32x4 ring[4][4][128];
            unsigned short sub[NBLK][4][32];
            unsigned short meta[NBLK][4];
            unsigned packed[NBLK];
            unsigned done[NBLK];
            unsigned lead;
        } s;
    };
};

__device__ __forceinline__ unsigned and_or(unsigned a, unsigned m, unsigned c) { return (a & m) | c; }

__device__ __forceinline__ unsigned wave_max(unsigned v) {
    v = max(v, (unsigned)__builtin_amdgcn_update_dpp(0, (int)v, 0x111, 0xf, 0xf, true));
    v = max(v, (unsigned)__builtin_amdgcn_update_dpp(0, (int)v, 0x112, 0xf, 0xf, true));
    v = max(v, (unsigned)__builtin_amdgcn_update_dpp(0, (int)v, 0x114, 0xf, 0xf, true));
    v = max(v, (unsigned)__builtin_amdgcn_update_dpp(0, (int)v, 0x118, 0xf, 0xf, true));
    v = max(v, (unsigned)__builtin_amdgcn_update_dpp(0, (int)v, 0x142, 0xa, 0xf, true));
    v = max(v, (unsigned)__builtin_amdgcn_update_dpp(0, (int)v, 0x143, 0xc, 0xf, true));
    return (unsigned)__builtin_amdgcn_readlane((int)v, 63);
}

__device__ __forceinline__ void fetch_unit(const uint8_t* src, unsigned lds) {
    unsigned keep;
    asm volatile(
        "s_mov_b32 %0, m0\n\t"
        "s_mov_b32 m0, %3\n\t"
        "s_nop 0\n\t"
        "global_load_lds_dwordx4 %1, off\n\t"
        "s_add_u32 m0, %3, 1024\n\t"
        "s_nop 0\n\t"
        "global_load_lds_dwordx4 %2, off\n\t"
        "s_mov_b32 m0, %0"
        : "=&s"(keep) : "v"(src), "v"(src + 64), "s"(lds) : "memory", "scc");
}

struct Packed {
    u32x4 lo;
    u32x3 hi;
    unsigned d7;
    bool raw;
};

constexpr unsigned SUB_CAP = 31;
__device__ __forceinline__ void pack_unit(const u32x4* slot, Smem& sm, int t, int s, int l, unsigned& a, bool& refresh, Packed& o) {
    const u32x4 R0 = slot[l], R1 = slot[64 + l];
    const unsigned f[8] = {R0[0], R0[1], R0[2], R0[3], R1[0], R1[1], R1[2], R1[3]};
    if (refresh) {
        const unsigned e0 = max(f[0] & 0x7Fu, f[4] & 0x7Fu), e1 = max((f[0] >> 8) & 0x7Fu, (f[4] >> 8) & 0x7Fu);
        const unsigned e2 = max((f[0] >> 16) & 0x7Fu, (f[4] >> 16) & 0x7Fu), e3 = max((f[0] >> 24) & 0x7Fu, (f[4] >> 24) & 0x7Fu);
        const unsigned mx = wave_max(max(max(e0, e1), max(e2, e3)));
        a = mx > 63u ? mx - 63u : 0u;
    }
    const unsigned na = (128u - a) * 0x01010101u;
    unsigned tt[8], m = 0;
#pragma unroll
    for (int j = 0; j < 8; ++j) tt[j] = (f[j] & 0x7F7F7F7Fu) + na;
#pragma unroll
    for (int j = 0; j < 7; ++j) m = and_or(tt[j] >> (6 - j), 0x01010101u << j, m);
    m = and_or(tt[7] << 1, 0x80808080u, m);
    unsigned short* l16 = sm.s.sub[t][s];
    const unsigned tag = (unsigned)((s << 11) | (l << 5));
    unsigned n = 0, mm = m;
    for (;;) {
        const unsigned long long bal = __ballot(mm != 0u);
        if (bal == 0ull || n > SUB_CAP) break;
        const unsigned idx = n + __builtin_amdgcn_mbcnt_hi((unsigned)(bal >> 32), __builtin_amdgcn_mbcnt_lo((unsigned)bal, 0u));
        if (mm != 0u) l16[min(idx, SUB_CAP)] = (unsigned short)(tag | (unsigned)(__ffs(mm) - 1));
        mm &= mm - 1u;
        n += (unsigned)__popcll(bal);
    }
    o.raw = n > SUB_CAP;
    refresh = n > 27u;
    {
        unsigned long long sv;
        const unsigned meta_lds = (unsigned)(uintptr_t)&sm.s.meta[t][s], cnt_lds = (unsigned)(uintptr_t)&sm.s.packed[t];
        asm volatile("s_mov_b64 %0, exec\n\ts_mov_b64 exec, 1\n\tds_write_b16 %1, %2\n\tds_add_u32 %3, %4\n\ts_mov_b64 exec, %0"
                     : "=&s"(sv) : "v"(meta_lds), "v"((a << 8) | (o.raw ? 0xFFu : n)), "v"(cnt_lds), "v"(1u) : "memory");
    }
    const unsigned p7 = and_or(tt[7], 0x3F3F3F3Fu, f[7] & 0x80808080u);
    unsigned d[7];
#pragma unroll
    for (int j = 0; j < 7; ++j) {
        const unsigned sp = j < 6 ? (p7 << (6 - j)) : (p7 >> 1);
        d[j] = and_or(sp, 0x40404040u, and_or(tt[j], 0x3F3F3F3Fu, f[j] & 0x80808080u));
    }
    if (o.raw) {
        o.lo = R0;
        o.hi = u32x3{f[4], f[5], f[6]};
    } else {
        o.lo = u32x4{d[0], d[1], d[2], d[3]};
        o.hi = u32x3{d[4], d[5], d[6]};
    }
    o.d7 = f[7];
}

__device__ __forceinline__ void store_unit(const Packed& o, uint8_t* piece, int s, int l) {
    asm volatile(
        "global_store_dwordx4 %0, %1, %4 sc0 sc1\n\t"
        "global_store_dwordx3 %2, %3, %4 sc0 sc1\n\t"
        "s_nop 1"
        :: "v"(s * 1024 + l * 16), "v"(o.lo), "v"(HI_OFF + s * 768 + l * 12), "v"(o.hi), "s"(piece) : "memory");
    if (o.raw)
        asm volatile("global_store_dword %0, %1, %2 sc0 sc1\n\ts_nop 1"
                     :: "v"(RAW7_OFF + s * 256 + l * 4), "v"(o.d7), "s"(piece) : "memory");
}

__device__ __forceinline__ bool lds_wait(const unsigned* word, unsigned value, unsigned long long t0) {
    unsigned itn = 0;
    while (__hip_atomic_load(word, __ATOMIC_RELAXED, __HIP_MEMORY_SCOPE_WORKGROUP) < value) {
        if ((++itn & 1023u) == 0u && __builtin_amdgcn_s_memrealtime() - t0 > DEADLINE_TICKS) return false;
        __builtin_amdgcn_s_sleep(1);
    }
    asm volatile("" ::: "memory");
    return true;
}

__device__ __forceinline__ void pusher(const Params& p, Smem& sm, int dd, int w) {
    const int tid = threadIdx.x;
    const int s = __builtin_amdgcn_readfirstlane(tid >> 6);
    const int l = tid & 63, r = l & 15, q = l >> 4;
    const unsigned long long t_entry = __builtin_amdgcn_s_memrealtime();
    const uint8_t* const px = p.x;
    uint8_t* const me = p.me;
    uint8_t* const pw = p.peer[dd];
    const int rank = p.rank;
    asm volatile("" :: "s"(px), "s"(me), "s"(pw), "s"(rank));
    const uint8_t* src = px + (size_t)r * KD + 512 * w + 128 * (s & 3) + 16 * q;
    u32x4* ring = &sm.s.ring[s & 3][0][0];
    const unsigned ring_lds = (unsigned)__builtin_amdgcn_readfirstlane((int)(unsigned)(uintptr_t)ring);
    unsigned ticket;
    {
        unsigned long long sv;
        asm volatile("s_mov_b64 %1, exec\n\ts_mov_b64 exec, 1\n\tglobal_atomic_add %0, %2, %3, off sc0\n\ts_mov_b64 exec, %1"
                     : "=&v"(ticket), "=&s"(sv)
                     : "v"((unsigned*)(me + CNT_OFF + (size_t)blockIdx.x * 128) + s), "v"(1u) : "memory");
    }
    if (__builtin_expect(s < 4, 1)) {
        fetch_unit(src, ring_lds);
        fetch_unit(src + BLK_BYTES, ring_lds + 2048u);
    }
    if (tid < NBLK) { sm.s.packed[tid] = 0u; sm.s.done[tid] = 0u; }
    if (tid == 0) sm.s.lead = 0u;
    asm volatile("s_waitcnt lgkmcnt(0)" ::: "memory");
    __builtin_amdgcn_s_barrier();
    asm volatile("s_waitcnt vmcnt(0)" : "+v"(ticket) :: "memory");
    if (s > 4) return;
    const unsigned call = (unsigned)__builtin_amdgcn_readfirstlane((int)ticket) + 1u;
    if (s == 4) {
        unsigned* flag = (unsigned*)(pw + FLAG_OFF + (size_t)(rank * 8 + w) * 128);
        const unsigned base = (call - 1u) * NBLK;
        uint8_t* dst = (uint8_t*)uniform_ptr(pw + STAGE_OFF + (size_t)(call & 1u) * SLOT_BYTES + (size_t)rank * SSHARD_BYTES +
                                             w * PIECE_BYTES + LIST_OFF);
        {
            const unsigned long long M40 = (1ull << 40) - 1ull;
            const unsigned long long key = ((unsigned long long)call << 40) | (M40 - (t_entry & M40));
            unsigned long long old = 0;
            if (l == 0) old = __hip_atomic_fetch_max((unsigned long long*)(me + T0_OFF + (size_t)dd * 128), key, __ATOMIC_RELAXED, __HIP_MEMORY_SCOPE_AGENT);
            if (l == 0 && old > key && (old >> 40) == (key >> 40)) {
                const unsigned long long d = old - key;
                __hip_atomic_store(&sm.s.lead, d < 150ull ? (unsigned)d : 150u, __ATOMIC_RELAXED, __HIP_MEMORY_SCOPE_WORKGROUP);
            }
            WAITV(0);
        }
        const unsigned long long t0 = __builtin_amdgcn_s_memrealtime();
#define PUBLISH(t) do { unsigned long long sv_; \
            asm volatile("s_mov_b64 %0, exec\n\ts_mov_b64 exec, 1\n\tglobal_atomic_umax %1, %2, off sc1\n\ts_mov_b64 exec, %0" \
                         : "=&s"(sv_) : "v"(flag), "v"(base + (unsigned)(t) + 1u) : "memory"); } while (0)
#pragma unroll 1
        for (int t = 0; t < NBLK; ++t) {
            if (!lds_wait(&sm.s.packed[t], 4u, t0)) return;
            const bool plain = plain_block(t);
            unsigned v = 0, nv = 0;
            if (!plain) {
                const unsigned m01 = (unsigned)__builtin_amdgcn_readfirstlane((int)((const unsigned*)sm.s.meta[t])[0]);
                const unsigned m23 = (unsigned)__builtin_amdgcn_readfirstlane((int)((const unsigned*)sm.s.meta[t])[1]);
                const unsigned c0 = m01 & 0xFFu, c1 = (m01 >> 16) & 0xFFu, c2 = m23 & 0xFFu, c3 = (m23 >> 16) & 0xFFu;
                const unsigned rawb = (c0 == 0xFFu ? 1u : 0u) | (c1 == 0xFFu ? 2u : 0u) | (c2 == 0xFFu ? 4u : 0u) | (c3 == 0xFFu ? 8u : 0u);
                const unsigned p1 = c0 == 0xFFu ? 0u : c0, p2 = p1 + (c1 == 0xFFu ? 0u : c1), p3 = p2 + (c2 == 0xFFu ? 0u : c2);
                nv = p3 + (c3 == 0xFFu ? 0u : c3);
                const unsigned short* sub = &sm.s.sub[t][0][0];
#pragma unroll
                for (int h = 0; h < 2; ++h) {
                    const unsigned i = 2u * (unsigned)l - 4u + (unsigned)h;
                    unsigned d = i >= p1 ? 32u - p1 : 0u;
                    d = i >= p2 ? 64u - p2 : d;
                    d = i >= p3 ? 96u - p3 : d;
                    v |= (unsigned)sub[(i + d) & 127u] << (16 * h);
                }
                if (l == 0) v = nv | (rawb << 16);
                if (l == 1) v = ((m01 >> 8) & 0xFFu) | ((m01 >> 24) << 8) | (((m23 >> 8) & 0xFFu) << 16) | ((m23 >> 24) << 24);
            }
            const unsigned nl = plain ? 1u : 2u + (nv + 1u) / 2u;
            const unsigned long long em = nl >= 64u ? ~0ull : ((1ull << nl) - 1ull);
            unsigned long long sv;
            asm volatile("s_mov_b64 %0, exec\n\ts_mov_b64 exec, %4\n\tglobal_store_dword %1, %2, %3 sc0 sc1\n\ts_mov_b64 exec, %0"
                         : "=&s"(sv) : "v"(l * 4 + (plain ? PIECE_BYTES - LIST_OFF - 4 : 0)), "v"(v), "s"(dst + (size_t)t * SBLK_BYTES), "s"(em) : "memory");
            if (t >= 2) {
                if (!lds_wait(&sm.s.done[t - 2], 4u, t0)) return;
                if (t >= 4) WAITV(4); else WAITV(2);
                PUBLISH(t - 2);
            }
        }
        if (!lds_wait(&sm.s.done[NBLK - 2], 4u, t0)) return;
        WAITV(3);
        PUBLISH(NBLK - 2);
        if (!lds_wait(&sm.s.done[NBLK - 1], 4u, t0)) return;
        WAITV(2);
        PUBLISH(NBLK - 1);
#undef PUBLISH
        return;
    }
    uint8_t* dst = (uint8_t*)uniform_ptr(pw + STAGE_OFF + (size_t)(call & 1u) * SLOT_BYTES + (size_t)rank * SSHARD_BYTES +
                                         w * PIECE_BYTES);
    const unsigned long long tstart = __builtin_amdgcn_s_memrealtime();
    unsigned win_a = 0;
    bool win_refresh = true;
#define LDS_COUNT(word) do { unsigned long long sv_; \
        asm volatile("s_mov_b64 %0, exec\n\ts_mov_b64 exec, 1\n\tds_add_u32 %1, %2\n\ts_mov_b64 exec, %0" \
                     : "=&s"(sv_) : "v"((unsigned)(uintptr_t)&(word)), "v"(1u) : "memory"); } while (0)
#define SIGNAL(t) LDS_COUNT(sm.s.done[t])
#pragma unroll 1
    for (int t = 0; t < NBLK; ++t) {
        if (t >= 2) WAITV(6);
        Packed o;
        const bool plain = plain_block(t);
        u32x4 r0, r1;
        if (plain) {
            r0 = ring[(t & 3) * 128 + l];
            r1 = ring[(t & 3) * 128 + 64 + l];
            asm volatile("s_waitcnt lgkmcnt(0)" : "+v"(r0), "+v"(r1) :: "memory");
            LDS_COUNT(sm.s.packed[t]);
        } else {
            pack_unit(ring + (t & 3) * 128, sm, t, s, l, win_a, win_refresh, o);
        }
        const int tn = t + 2 < NBLK ? t + 2 : t;
        fetch_unit(src + (size_t)tn * BLK_BYTES, ring_lds + (unsigned)((t + 2) & 3) * 2048u);
        if (t >= 2) { WAITV(6); SIGNAL(t - 2); }
        {
            const int nplain = (t < PLAIN_HEAD ? t : PLAIN_HEAD) + (t > NBLK - PLAIN_TAIL ? t - (NBLK - PLAIN_TAIL) : 0);
            const unsigned lead = __hip_atomic_load(&sm.s.lead, __ATOMIC_RELAXED, __HIP_MEMORY_SCOPE_WORKGROUP);
            const unsigned long long due = tstart - (unsigned)__builtin_amdgcn_readfirstlane((int)lead) + (((unsigned long long)(8 * t + nplain) * PACE) >> 7);
            while (__builtin_amdgcn_s_memrealtime() < due) __builtin_amdgcn_s_sleep(1);
        }
        if (plain)
            asm volatile(
                "global_store_dwordx4 %0, %1, %3 sc0 sc1\n\t"
                "global_store_dwordx4 %0, %2, %3 offset:1024 sc0 sc1\n\t"
                "s_nop 1"
                :: "v"(s * 2048 + l * 16), "v"(r0), "v"(r1), "s"(dst + (size_t)t * SBLK_BYTES) : "memory");
        else
            store_unit(o, dst + (size_t)t * SBLK_BYTES, s, l);
    }
    WAITV(4); SIGNAL(NBLK - 2);
    WAITV(0); SIGNAL(NBLK - 1);
#undef SIGNAL
#undef LDS_COUNT
}

__device__ __forceinline__ void transpose4(unsigned a, unsigned b, unsigned c, unsigned d,
                                           unsigned& o0, unsigned& o1, unsigned& o2, unsigned& o3) {
    const unsigned t0 = __builtin_amdgcn_perm(b, a, 0x05010400u);
    const unsigned t1 = __builtin_amdgcn_perm(b, a, 0x07030602u);
    const unsigned t2 = __builtin_amdgcn_perm(d, c, 0x05010400u);
    const unsigned t3 = __builtin_amdgcn_perm(d, c, 0x07030602u);
    o0 = __builtin_amdgcn_perm(t2, t0, 0x05040100u);
    o1 = __builtin_amdgcn_perm(t2, t0, 0x07060302u);
    o2 = __builtin_amdgcn_perm(t3, t1, 0x05040100u);
    o3 = __builtin_amdgcn_perm(t3, t1, 0x07060302u);
}

__device__ __forceinline__ i32x8 join8(u32x4 lo, u32x4 hi) {
    i32x8 v;
    v[0] = lo[0]; v[1] = lo[1]; v[2] = lo[2]; v[3] = lo[3];
    v[4] = hi[0]; v[5] = hi[1]; v[6] = hi[2]; v[7] = hi[3];
    return v;
}

__device__ __forceinline__ i32x8 unpack_unit(u32x4 lo, u32x3 hi, unsigned m, unsigned A) {
    const unsigned d[7] = {lo[0], lo[1], lo[2], lo[3], hi[0], hi[1], hi[2]};
    i32x8 f;
    unsigned g = (d[6] << 1) & 0x80808080u;
#pragma unroll
    for (int j = 0; j < 6; ++j) g = and_or(d[j] >> (6 - j), 0x01010101u << j, g);
#pragma unroll
    for (int j = 0; j < 7; ++j) f[j] = (int)(((d[j] & 0xBFBFBFBFu) + A) ^ ((m << (6 - j)) & 0x40404040u));
    f[7] = (int)((g + A) ^ ((m >> 1) & 0x40404040u));
    return f;
}

__device__ __forceinline__ void gemm_wg(const Params& p, Smem& sm, int g, int j) {
    const int tid = threadIdx.x, w = __builtin_amdgcn_readfirstlane(tid >> 6), l = tid & 63, r = l & 15, q = l >> 4;
    if (tid < 2) sm.g.bad[tid] = 0u;
    if (tid < 128) (&sm.g.poll[0][0])[tid] = 0u;
    for (int i = 0; i < 8; ++i) __builtin_amdgcn_s_sleep(2);
    i32x8 Bf[4][4];
    {
        const uint8_t* wp = p.w + (size_t)(512 * w + 16 * q) * NS + 64 * j + 4 * r;
#pragma unroll
        for (int s = 0; s < 4; ++s) {
            unsigned t[32];
#pragma unroll
            for (int h = 0; h < 2; ++h)
#pragma unroll
                for (int k = 0; k < 16; ++k)
                    t[h * 16 + k] = *(const unsigned*)(wp + (size_t)(128 * s + 64 * h + k) * NS);
#pragma unroll
            for (int jj = 0; jj < 8; ++jj) {
                unsigned o0, o1, o2, o3;
                transpose4(t[4 * jj], t[4 * jj + 1], t[4 * jj + 2], t[4 * jj + 3], o0, o1, o2, o3);
                Bf[s][0][jj] = o0; Bf[s][1][jj] = o1; Bf[s][2][jj] = o2; Bf[s][3][jj] = o3;
            }
        }
    }
    const float scale = p.sx[0] * p.sw[0];
    const unsigned call = draw_call(p);
    __syncthreads();
    unsigned* tmo = (unsigned*)(p.me + TMO_OFF);
    const uint8_t* stage = p.me + STAGE_OFF + (size_t)(call & 1u) * SLOT_BYTES + w * PIECE_BYTES;
    float* outp = p.out + (size_t)(tid >> 5) * NS + 64 * j + (tid & 31) * 2;
    const int nloc = g >= 16 ? 2 : 1;
    const int nrem = g < 8 ? 10 : 9;
    bool bad = false;

    auto finish = [&](const f32x4 (&acc)[4], int buf, int row0) {
#pragma unroll
        for (int t = 0; t < 4; ++t) {
            f32x4 v = f32x4{acc[0][t], acc[1][t], acc[2][t], acc[3][t]};
            asm volatile("" : "+v"(v));
            sm.g.red[buf][w][4 * q + t][r] = v;
        }
        if (bad) sm.g.bad[buf] = 1u;
        __syncthreads();
        const bool anybad = sm.g.bad[buf] != 0u;
        const f32x2* rp = (const f32x2*)&sm.g.red[buf][0][0][0] + tid;
        f32x2 v = rp[0];
#pragma unroll
        for (int k = 1; k < 8; ++k) v += rp[k * 512];
        v *= scale;
        v[0] = v[0] < 0.f ? 0.f : v[0];
        v[1] = v[1] < 0.f ? 0.f : v[1];
        if (anybad) v = f32x2{__builtin_nanf(""), __builtin_nanf("")};
        asm volatile("global_store_dwordx2 %0, %1, off sc0 sc1\n\ts_nop 0" :: "v"(outp + (size_t)row0 * NS), "v"(v) : "memory");
    };

    auto item_plain = [&](int k) -> bool { return plain_block((g + NGROUP * k) / 7); };
    auto item = [&](int k, const uint8_t*& pc, const unsigned*& flag, unsigned& seq, int& row0) {
        const int n = g + NGROUP * k;
        const int i = n % 7, b = n / 7;
        const unsigned srcr = (unsigned)(p.rank + 1 + i) & 7u;
        pc = uniform_ptr(stage + (size_t)srcr * SSHARD_BYTES + (size_t)b * SBLK_BYTES);
        flag = (const unsigned*)uniform_ptr(p.me + FLAG_OFF + (size_t)(srcr * 8 + w) * 128);
        seq = (call - 1u) * NBLK + b + 1u;
        row0 = (int)srcr * MS + 16 * b;
    };
    const unsigned poll_lds = (unsigned)__builtin_amdgcn_readfirstlane((int)(unsigned)(uintptr_t)&sm.g.poll[w][0]);
    auto poll = [&](int k, const unsigned* flag) {
        unsigned keep; unsigned long long sv;
        asm volatile(
            "s_mov_b32 %0, m0\n\ts_mov_b64 %1, exec\n\ts_mov_b32 m0, %4\n\ts_mov_b64 exec, 1\n\t"
            "global_load_lds_dword %2, %3 sc0 sc1\n\t"
            "s_mov_b64 exec, %1\n\ts_mov_b32 m0, %0"
            : "=&s"(keep), "=&s"(sv) : "v"(0u), "s"(flag), "s"(poll_lds + 4u * (unsigned)k) : "memory");
    };
    auto polled = [&](int k) -> unsigned {
        return (unsigned)__builtin_amdgcn_readfirstlane((int)__hip_atomic_load(&sm.g.poll[w][k], __ATOMIC_RELAXED, __HIP_MEMORY_SCOPE_WORKGROUP));
    };
    const unsigned* ab = (const unsigned*)&sm.g.abuf[w][0];
    const unsigned ab_lds = (unsigned)__builtin_amdgcn_readfirstlane((int)(unsigned)(uintptr_t)ab);
    auto fetch_piece = [&](const uint8_t* pc, bool plain) {
        unsigned keep; unsigned long long sv;
        asm volatile(
            "s_mov_b32 %0, m0\n\t"
            "s_mov_b32 m0, %5\n\ts_nop 0\n\t"
            "global_load_lds_dwordx4 %2, %3 sc0 sc1\n\t"
            "global_load_lds_dwordx4 %2, %3 offset:1024 sc0 sc1\n\t"
            "global_load_lds_dwordx4 %2, %3 offset:2048 sc0 sc1\n\t"
            "global_load_lds_dwordx4 %2, %3 offset:3072 sc0 sc1\n\t"
            "s_add_u32 m0, %5, 4096\n\ts_nop 0\n\t"
            "global_load_lds_dwordx4 %2, %4 sc0 sc1\n\t"
            "global_load_lds_dwordx4 %2, %4 offset:1024 sc0 sc1\n\t"
            "global_load_lds_dwordx4 %2, %4 offset:2048 sc0 sc1\n\t"
            "s_mov_b64 %1, exec\n\ts_mov_b64 exec, %6\n\t"
            "global_load_lds_dwordx4 %2, %4 offset:3072 sc0 sc1\n\t"
            "s_mov_b64 exec, %1\n\t"
            "s_mov_b32 m0, %0"
            : "=&s"(keep), "=&s"(sv) : "v"(l * 16), "s"(pc), "s"(pc + 4096), "s"(ab_lds), "s"(plain ? ~0ull : 0xffffull) : "memory", "scc");
    };
    {
        const uint8_t* pc; const unsigned* flag; unsigned seq; int row0;
        item(0, pc, flag, seq, row0); poll(0, flag);
        item(1, pc, flag, seq, row0); poll(1, flag);
    }

    for (int it = 0; it < nloc; ++it) {
        const int lb = g + 8 * it;
        const uint8_t* src = p.x + (size_t)(16 * lb + r) * KD + 512 * w + 16 * q;
        u32x4 Af[8];
        asm volatile(
            "global_load_dwordx4 %0, %8, off\n\t"
            "global_load_dwordx4 %1, %8, off offset:64\n\t"
            "global_load_dwordx4 %2, %8, off offset:128\n\t"
            "global_load_dwordx4 %3, %8, off offset:192\n\t"
            "global_load_dwordx4 %4, %8, off offset:256\n\t"
            "global_load_dwordx4 %5, %8, off offset:320\n\t"
            "global_load_dwordx4 %6, %8, off offset:384\n\t"
            "global_load_dwordx4 %7, %8, off offset:448\n\t"
            "s_waitcnt vmcnt(0)"
            : "=&v"(Af[0]), "=&v"(Af[1]), "=&v"(Af[2]), "=&v"(Af[3]), "=&v"(Af[4]), "=&v"(Af[5]), "=&v"(Af[6]), "=&v"(Af[7])
            : "v"(src) : "memory");
        f32x4 acc[4];
#pragma unroll
        for (int i = 0; i < 4; ++i) acc[i] = f32x4{0.f, 0.f, 0.f, 0.f};
#pragma unroll
        for (int s = 0; s < 4; ++s) {
            const i32x8 a = join8(Af[2 * s], Af[2 * s + 1]);
#pragma unroll
            for (int i = 0; i < 4; ++i)
                acc[i] = __builtin_amdgcn_mfma_scale_f32_16x16x128_f8f6f4(a, Bf[s][i], acc[i], 0, 0, 0, 0, 0, 0);
        }
        finish(acc, it & 1, p.rank * MS + 16 * lb);
    }

    unsigned* mk = sm.g.mask[w];
    bool issued = false;
#pragma unroll 1
    for (int k = 0; k < nrem; ++k) {
        const uint8_t* pc; const unsigned* flag; unsigned seq; int row0;
        item(k, pc, flag, seq, row0);
        const bool was_issued = issued;
        if (issued) {
            WAITV(2);
        } else {
            if (!bad && polled(k) < seq) {
                const unsigned long long t0 = __builtin_amdgcn_s_memrealtime();
                unsigned itn = 0;
                for (;;) {
                    poll(k, flag);
                    __builtin_amdgcn_s_sleep(3);
                    if (polled(k) >= seq) break;
                    if ((++itn & 15u) == 0u &&
                        (ld_agent(tmo) == call || __builtin_amdgcn_s_memrealtime() - t0 > DEADLINE_TICKS)) {
                        if (l == 0) __hip_atomic_store(tmo, call, __ATOMIC_RELAXED, __HIP_MEMORY_SCOPE_AGENT);
                        bad = true;
                        break;
                    }
                }
            }
            if (!item_plain(k)) {
                fetch_piece(pc, false);
                WAITV(0);
            }
        }
        const bool direct = !was_issued && item_plain(k);
        const bool last = k + 1 == nrem;
        const uint8_t* pcn = pc; const unsigned* flagn = flag; unsigned seqn = 0xFFFFFFFFu; int row0n;
        if (!last) {
            item(k + 1, pcn, flagn, seqn, row0n);
            poll(k + 1, flagn);
        }
        f32x4 acc[4];
#pragma unroll
        for (int i2 = 0; i2 < 4; ++i2) acc[i2] = f32x4{0.f, 0.f, 0.f, 0.f};
        if (direct) {
            u32x4 fr[8];
            const uint8_t* pc4 = pc + 4096;
            asm volatile(
                "global_load_dwordx4 %0, %8, %9 sc0 sc1\n\t"
                "global_load_dwordx4 %1, %8, %9 offset:1024 sc0 sc1\n\t"
                "global_load_dwordx4 %2, %8, %9 offset:2048 sc0 sc1\n\t"
                "global_load_dwordx4 %3, %8, %9 offset:3072 sc0 sc1\n\t"
                "global_load_dwordx4 %4, %8, %10 sc0 sc1\n\t"
                "global_load_dwordx4 %5, %8, %10 offset:1024 sc0 sc1\n\t"
                "global_load_dwordx4 %6, %8, %10 offset:2048 sc0 sc1\n\t"
                "global_load_dwordx4 %7, %8, %10 offset:3072 sc0 sc1"
                : "=&v"(fr[0]), "=&v"(fr[1]), "=&v"(fr[2]), "=&v"(fr[3]), "=&v"(fr[4]), "=&v"(fr[5]), "=&v"(fr[6]), "=&v"(fr[7])
                : "v"(l * 16), "s"(pc), "s"(pc4) : "memory");
#define KSTEP(s, nw) { \
            asm volatile("s_waitcnt vmcnt(" #nw ")" : "+v"(fr[2 * s]), "+v"(fr[2 * s + 1]) :: "memory"); \
            const i32x8 a = join8(fr[2 * s], fr[2 * s + 1]); \
            _Pragma("unroll") for (int i2 = 0; i2 < 4; ++i2) \
                acc[i2] = __builtin_amdgcn_mfma_scale_f32_16x16x128_f8f6f4(a, Bf[s][i2], acc[i2], 0, 0, 0, 0, 0, 0); \
            __builtin_amdgcn_sched_barrier(0); }
            KSTEP(0, 6) KSTEP(1, 4) KSTEP(2, 2) KSTEP(3, 0)
#undef KSTEP
            issued = false;
        } else if (item_plain(k)) {
            u32x4 fr[8];
#pragma unroll
            for (int s = 0; s < 8; ++s) fr[s] = *(const u32x4*)(ab + s * 256 + l * 4);
            asm volatile("s_waitcnt lgkmcnt(0)" : "+v"(fr[0]), "+v"(fr[1]), "+v"(fr[2]), "+v"(fr[3]), "+v"(fr[4]), "+v"(fr[5]),
                         "+v"(fr[6]), "+v"(fr[7]) :: "memory");
            issued = !last && !bad && polled(k + 1) >= seqn;
            if (issued) fetch_piece(pcn, item_plain(k + 1));
#pragma unroll
            for (int s = 0; s < 4; ++s) {
                const i32x8 a = join8(fr[2 * s], fr[2 * s + 1]);
#pragma unroll
                for (int i2 = 0; i2 < 4; ++i2)
                    acc[i2] = __builtin_amdgcn_mfma_scale_f32_16x16x128_f8f6f4(a, Bf[s][i2], acc[i2], 0, 0, 0, 0, 0, 0);
            }
        } else {
        *(u32x4*)&mk[l * 4] = u32x4{0u, 0u, 0u, 0u};
        unsigned ex = ab[LIST_OFF / 4 + l];
        u32x4 lo[4];
        u32x3 hi[4];
#pragma unroll
        for (int s = 0; s < 4; ++s) {
            lo[s] = *(const u32x4*)(ab + s * 256 + l * 4);
            const unsigned* h = ab + HI_OFF / 4 + s * 192 + l * 3;
            hi[s] = u32x3{h[0], h[1], h[2]};
        }
        asm volatile("s_waitcnt lgkmcnt(0)" : "+v"(ex), "+v"(lo[0]), "+v"(lo[1]), "+v"(lo[2]), "+v"(lo[3]),
                     "+v"(hi[0]), "+v"(hi[1]), "+v"(hi[2]), "+v"(hi[3]) :: "memory");
        issued = !last && !bad && polled(k + 1) >= seqn;
        if (issued) fetch_piece(pcn, item_plain(k + 1));
        const unsigned h0 = (unsigned)__builtin_amdgcn_readlane((int)ex, 0), h1 = (unsigned)__builtin_amdgcn_readlane((int)ex, 1);
        const unsigned nv = h0 & 0xFFFFu, rawb = h0 >> 16;
        const unsigned i0 = 2u * (unsigned)l - 4u;
        if (i0 < nv) atomicOr(&mk[(ex >> 5) & 255u], 1u << (ex & 31u));
        if (i0 + 1u < nv && l >= 2) atomicOr(&mk[(ex >> 21) & 255u], 1u << ((ex >> 16) & 31u));
        unsigned mv[4];
#pragma unroll
        for (int s = 0; s < 4; ++s) mv[s] = mk[s * 64 + l];
        if (rawb == 0u) {
            i32x8 a = unpack_unit(lo[0], hi[0], mv[0], (h1 & 0xFFu) * 0x01010101u);
#define SB() __builtin_amdgcn_sched_barrier(0)
#define UF(j) an[j] = (int)(((d[j] & 0xBFBFBFBFu) + A) ^ ((m << (6 - j)) & 0x40404040u))
#define UG(j) gg = and_or(d[j] >> (6 - j), 0x01010101u << j, gg)
#define MM(s, i) acc[i] = __builtin_amdgcn_mfma_scale_f32_16x16x128_f8f6f4(a, Bf[s][i], acc[i], 0, 0, 0, 0, 0, 0)
#pragma unroll
            for (int s = 0; s < 3; ++s) {
                const unsigned d[7] = {lo[s + 1][0], lo[s + 1][1], lo[s + 1][2], lo[s + 1][3], hi[s + 1][0], hi[s + 1][1], hi[s + 1][2]};
                const unsigned m = mv[s + 1], A = ((h1 >> (8 * (s + 1))) & 0xFFu) * 0x01010101u;
                i32x8 an;
                unsigned gg;
                SB(); MM(s, 0); SB();
                UF(0); UF(1); gg = (d[6] << 1) & 0x80808080u; UG(0);
                SB(); MM(s, 1); SB();
                UF(2); UF(3); UG(1); UG(2);
                SB(); MM(s, 2); SB();
                UF(4); UF(5); UG(3); UG(4);
                SB(); MM(s, 3); SB();
                UF(6); UG(5);
                an[7] = (int)((gg + A) ^ ((m >> 1) & 0x40404040u));
                a = an;
            }
            SB(); MM(3, 0); MM(3, 1); MM(3, 2); MM(3, 3); SB();
#undef SB
#undef UF
#undef UG
#undef MM
        } else {
#pragma unroll
            for (int s = 0; s < 4; ++s) {
                i32x8 a;
                if ((rawb >> s) & 1u) {
                    unsigned d7;
                    asm volatile("global_load_dword %0, %1, %2 sc0 sc1\n\ts_waitcnt vmcnt(0)"
                                 : "=&v"(d7) : "v"(RAW7_OFF + s * 256 + l * 4), "s"(pc) : "memory");
                    a[0] = lo[s][0]; a[1] = lo[s][1]; a[2] = lo[s][2]; a[3] = lo[s][3];
                    a[4] = hi[s][0]; a[5] = hi[s][1]; a[6] = hi[s][2]; a[7] = d7;
                } else {
                    a = unpack_unit(lo[s], hi[s], mv[s], ((h1 >> (8 * s)) & 0xFFu) * 0x01010101u);
                }
#pragma unroll
                for (int i2 = 0; i2 < 4; ++i2)
                    acc[i2] = __builtin_amdgcn_mfma_scale_f32_16x16x128_f8f6f4(a, Bf[s][i2], acc[i2], 0, 0, 0, 0, 0, 0);
            }
        }
        }
        if (!last) {
            if (!issued && !bad && polled(k + 1) >= seqn) { fetch_piece(pcn, item_plain(k + 1)); issued = true; }
            const int k2 = k + 2 < nrem ? k + 2 : k;
            const uint8_t* pc2; const unsigned* flag2; unsigned seq2; int row2;
            item(k2, pc2, flag2, seq2, row2);
            poll(k2, flag2);
        }
        finish(acc, (k + nloc) & 1, row0);
    }
}

__global__ void __launch_bounds__(NTHREADS, 2) ag_gemm_kernel(Params p) {
    __shared__ Smem sm;
    const int xcd = blockIdx.x & 7, slot = blockIdx.x >> 3;
    if (__builtin_expect(slot < 7, 1)) pusher(p, sm, slot, xcd);
    else if (slot >= 8) gemm_wg(p, sm, 3 * xcd + ((slot - 8) >> 3), (slot - 8) & 7);
}

}

extern "C" void kernel_launch(int rank, int nranks, ncclComm_t comm, hipStream_t stream,
                              void* const* d_in, const int* in_sizes, int n_in,
                              void* d_out, int out_size, const int* in_dtypes, int out_dtype,
                              void* const* d_out_all, void* const* d_ws_all, size_t ws_bytes) {
    (void)comm; (void)in_dtypes; (void)out_dtype; (void)d_out_all;
    if (nranks != NR || n_in != 4 || (unsigned)rank >= (unsigned)NR || ws_bytes < WS_NEED ||
        in_sizes[0] != MS * KD || in_sizes[1] != KD * NS || out_size != NR * MS * NS) {
        fprintf(stderr, "kernel_launch: unexpected problem shape\n");
        return;
    }
    Params p;
    p.x = (const uint8_t*)d_in[0];
    p.w = (const uint8_t*)d_in[1];
    p.sx = (const float*)d_in[2];
    p.sw = (const float*)d_in[3];
    p.out = (float*)d_out;
    p.me = (uint8_t*)d_ws_all[rank];
    for (int i = 0; i < 7; ++i) p.peer[i] = (uint8_t*)d_ws_all[(rank + 1 + i) % NR];
    p.rank = rank;
    p.pad = 0;
    hipLaunchKernelGGL(ag_gemm_kernel, dim3(NGRID), dim3(NTHREADS), 0, stream, p);
    hipError_t e = hipGetLastError();
    if (e != hipSuccess) fprintf(stderr, "launch error %s\n", hipGetErrorString(e));
}
```
